```python
import numpy as np

kernel_name = "gpu_gemm_a2a_m4096_k4096_n16384_bf16_reluqe4m3"


ABSMAX_RTOL = 0.09

COMPUTE_DTYPE = "bfloat16"
EPILOGUE = "relu_quant_e4m3"
IN_QUANT = None
OUT_QUANT = "e4m3"
IN_DTYPES = {"x": "bfloat16", "w_mat": "bfloat16"}
WORLD_SIZE = 8

def _to_bf16(x):
    u = np.ascontiguousarray(x, dtype=np.float32).view(np.uint32)
    return ((u + 0x7FFF + ((u >> 16) & 1)) & 0xFFFF0000).view(np.float32)

def setup_inputs():

    x = _to_bf16(np.random.randn(4096, 4096).astype(np.float32))
    w_mat = _to_bf16((np.random.randn(4096, 16384) / np.sqrt(4096)).astype(np.float32))
    return {"x": x, "w_mat": w_mat}

def _snap_fp8(x, mant_bits, max_abs, min_normal):
    a = np.abs(x.astype(np.float64))
    _m, _e = np.frexp(a)
    q = np.where(a >= min_normal, np.ldexp(1.0, _e - 1 - mant_bits),
                 min_normal * 2.0 ** -mant_bits)
    snapped = np.minimum(np.round(a / q) * q, max_abs)
    return (np.sign(x) * snapped).astype(np.float32)
def _quant_dequant(y, amax):
    scale = amax / 448.0
    q = _snap_fp8(y / scale, 3, 448.0, 2.0 ** -6)
    return (q * scale).astype(np.float32)
def _epilogue(y):
    y = np.maximum(y, 0.0)
    return _quant_dequant(y, np.abs(y).max())

def reference(x, w_mat):
    y = (x @ w_mat).astype(np.float32)
    return _epilogue(y)
```

```cpp
#include <hip/hip_runtime.h>
#include <rccl/rccl.h>
#include <cstdio>

#define HIP_CHECK(expr) do { hipError_t _e = (expr); if (_e != hipSuccess) { \
    fprintf(stderr, "HIP error %s at %s:%d\n", hipGetErrorString(_e), __FILE__, __LINE__); \
    return; } } while (0)

namespace {

typedef short bf16x8 __attribute__((ext_vector_type(8)));
typedef short s16x4 __attribute__((ext_vector_type(4)));
typedef float f32x4 __attribute__((ext_vector_type(4)));
typedef unsigned u32x4 __attribute__((ext_vector_type(4)));
typedef unsigned long long u64;
#define LAS __attribute__((address_space(3)))
#define GAS __attribute__((address_space(1)))

constexpr int NR = 8;
constexpr int MLOC = 512, KDIM = 4096, NDIM = 16384, NSH = NDIM / NR;
constexpr int BM = 256, BN = 128, BK = 64, NT = KDIM / BK;
constexpr int NWG = (MLOC / BM) * (NDIM / BN);
constexpr int X_SLOT = BM * BK * 2, W_SLOT = BK * BN * 2, XS = 3, WS = 4;
constexpr int W_BASE = XS * X_SLOT, LDS_BYTES = W_BASE + WS * W_SLOT;

constexpr size_t WS_CNT = 0;
constexpr size_t WS_TMO = WS_CNT + NWG * 128;
constexpr size_t WS_AGG = WS_TMO + 256;
constexpr size_t WS_GLB = WS_AGG + 2 * NWG * 8;
constexpr size_t WS_DONE = WS_GLB + 2 * 128;
constexpr size_t WS_END = WS_DONE + NR * 32 * 4;

constexpr u64 DEADLINE_TICKS = 50000000ull;

struct Params {
  const unsigned short* x;
  const unsigned short* w;
  unsigned char* out[NR];
  unsigned char* ws[NR];
  int rank;
  int pad;
};

#define BARRIER_L() asm volatile("s_waitcnt lgkmcnt(0)\n\ts_barrier" ::: "memory")
#define BARRIER_V(n) asm volatile("s_waitcnt vmcnt(" #n ")\n\ts_barrier" ::: "memory")

__device__ __forceinline__ bf16x8 ldx(LAS char* lds, unsigned addr, int imm) { return *(LAS bf16x8*)(lds + addr + imm); }
__device__ __forceinline__ bf16x8 ldw(LAS char* lds, unsigned addr, int imm) {
  s16x4 lo = __builtin_amdgcn_ds_read_tr16_b64_v4i16((LAS s16x4*)(lds + addr + imm));
  s16x4 hi = __builtin_amdgcn_ds_read_tr16_b64_v4i16((LAS s16x4*)(lds + addr + imm + 1024));
  return __builtin_shufflevector(lo, hi, 0, 1, 2, 3, 4, 5, 6, 7);
}
__device__ __forceinline__ float wave_max(float v) {
#pragma unroll
  for (int o = 32; o >= 1; o >>= 1) v = fmaxf(v, __shfl_xor(v, o));
  return v;
}
__device__ __forceinline__ unsigned wave_max_u(unsigned v) {
#pragma unroll
  for (int o = 32; o >= 1; o >>= 1) { unsigned t = (unsigned)__shfl_xor((int)v, o); v = v > t ? v : t; }
  return v;
}

__global__ __launch_bounds__(512, 2) void fused_gemm_a2a(const Params p) {
  __shared__ __attribute__((aligned(16))) char smem_[LDS_BYTES];
  LAS char* lds = (LAS char*)smem_;
  const int bid = blockIdx.x;
  const int xcd = bid & 7, idx = bid >> 3;
  const int panel = (idx >> 1) * 8 + xcd, mt = idx & 1;
  const unsigned dst = (unsigned)(panel >> 4);
  const int wid = __builtin_amdgcn_readfirstlane(threadIdx.x >> 6);
  const int lane = threadIdx.x & 63;
  const int rank = p.rank;
  unsigned char* myws = p.ws[rank];
  LAS float* red = (LAS float*)lds;

  if (wid >= 4) {
    const int lw = wid - 4;
    auto rx = __builtin_amdgcn_make_buffer_rsrc((void*)(p.x + (size_t)mt * BM * KDIM), 0, BM * KDIM * 2, 0x00020000);
    auto rw = __builtin_amdgcn_make_buffer_rsrc((void*)p.w, 0, KDIM * NDIM * 2, 0x00020000);
    const unsigned xv = (unsigned)((lane >> 3) * (KDIM * 2) + (((lane & 7) ^ (4 * (lw & 1) + (lane >> 4))) * 16));
    const unsigned xj0 = (lw >> 1) * 16 + (lw & 1);
    const unsigned wv = (unsigned)((lane >> 4) * (NDIM * 2) + (((lane & 15) ^ ((lane >> 4) + 8 * (lw >> 1))) * 16) + panel * (BN * 2));
    const unsigned wj0 = lw;
    auto issue_x = [&](int t, int slot) {
      const unsigned so = (t < NT) ? (unsigned)(t * BK * 2) : 0x80000000u;
#pragma unroll
      for (int jj = 0; jj < 8; ++jj) {
        const unsigned j = xj0 + 2 * jj;
        __builtin_amdgcn_raw_ptr_buffer_load_lds(rx, (LAS void*)(lds + slot * X_SLOT + j * 1024), 16, xv, so + j * 8 * (KDIM * 2), 0, 0);
      }
    };
    auto issue_w = [&](int t, int slot) {
      const unsigned so = (t < NT) ? (unsigned)(t * BK * (NDIM * 2)) : 0x80000000u;
#pragma unroll
      for (int jj = 0; jj < 4; ++jj) {
        const unsigned j = wj0 + 4 * jj;
        __builtin_amdgcn_raw_ptr_buffer_load_lds(rw, (LAS void*)(lds + W_BASE + slot * W_SLOT + j * 1024), 16, wv, so + j * 4 * (NDIM * 2), 0, 0);
      }
    };
    issue_w(0, 0); issue_x(0, 0); issue_w(1, 1); issue_x(1, 1); issue_w(2, 2);
    int xs = 2, ws = 3;
    for (int t = 0; t < NT; ++t) {
      BARRIER_V(16);
      issue_x(t + 2, xs); issue_w(t + 3, ws);
      xs = (xs == XS - 1) ? 0 : xs + 1; ws = (ws + 1) & (WS - 1);
    }
    BARRIER_V(0);
    GAS unsigned* tmo = (GAS unsigned*)(myws + WS_TMO);
    BARRIER_L();
    const unsigned call = __builtin_amdgcn_readfirstlane(((LAS unsigned*)red)[10]);
    const unsigned par = call & 1u;
    if (lw == 0) {
      const float m = fmaxf(fmaxf(red[0], red[1]), fmaxf(red[2], red[3]));
      if (lane == 0)
        __hip_atomic_store((GAS u64*)(myws + WS_AGG) + par * NWG + bid, ((u64)call << 32) | __float_as_uint(m), __ATOMIC_RELAXED, __HIP_MEMORY_SCOPE_AGENT);
      GAS u64* g = (GAS u64*)(myws + WS_GLB + par * 128);
      const u64 t0 = __builtin_amdgcn_s_memrealtime();
      unsigned mv = 0; bool bad = false;
      for (;;) {
        const u64 v = __hip_atomic_load(g + (lane & 7), __ATOMIC_RELAXED, __HIP_MEMORY_SCOPE_SYSTEM);
        const unsigned tv = __hip_atomic_load(tmo, __ATOMIC_RELAXED, __HIP_MEMORY_SCOPE_AGENT);
        mv = (unsigned)v;
        if (__builtin_amdgcn_readfirstlane(tv) == call) { bad = true; break; }
        if (__all((unsigned)(v >> 32) == call)) break;
        if (__builtin_amdgcn_s_memrealtime() - t0 > DEADLINE_TICKS) {
          if (lane == 0) __hip_atomic_store(tmo, call, __ATOMIC_RELAXED, __HIP_MEMORY_SCOPE_AGENT);
          bad = true; break;
        }
        __builtin_amdgcn_s_sleep(1);
      }
      unsigned am = mv;
#pragma unroll
      for (int o = 4; o >= 1; o >>= 1) { unsigned t = (unsigned)__shfl_xor((int)am, o); am = am > t ? am : t; }
      const float amax = __uint_as_float(am);
      const float scale = amax / 448.0f;
      float inv = amax > 0.f ? 1.0f / scale : 0.f;
      if (lane == 0) { red[8] = inv; red[9] = bad ? 1.f : 0.f; }
    } else if (lw == 1 && bid == 0) {
      GAS u64* a = (GAS u64*)(myws + WS_AGG) + par * NWG;
      const u64 t0 = __builtin_amdgcn_s_memrealtime();
      bool bad = false; unsigned mv = 0;
      for (;;) {
        u64 v[4];
#pragma unroll
        for (int i = 0; i < 4; ++i) v[i] = __hip_atomic_load(a + lane + 64 * i, __ATOMIC_RELAXED, __HIP_MEMORY_SCOPE_AGENT);
        bool ok = true; mv = 0;
#pragma unroll
        for (int i = 0; i < 4; ++i) { ok &= (unsigned)(v[i] >> 32) == call; mv = mv > (unsigned)v[i] ? mv : (unsigned)v[i]; }
        if (__all(ok)) break;
        if (__builtin_amdgcn_s_memrealtime() - t0 > DEADLINE_TICKS) {
          if (lane == 0) __hip_atomic_store(tmo, call, __ATOMIC_RELAXED, __HIP_MEMORY_SCOPE_AGENT);
          bad = true; break;
        }
      }
      mv = wave_max_u(mv);
      if (!bad && lane < NR)
        __hip_atomic_store((GAS u64*)(p.ws[lane] + WS_GLB + par * 128) + rank, ((u64)call << 32) | mv, __ATOMIC_RELAXED, __HIP_MEMORY_SCOPE_SYSTEM);
    }
    BARRIER_L();
    BARRIER_L();
    if (lw == 0) {
      if (lane == 0 && dst != (unsigned)rank && dst < (unsigned)NR)
        __hip_atomic_store((GAS unsigned*)(p.ws[dst] + WS_DONE) + rank * 32 + (panel & 15) * 2 + mt, call, __ATOMIC_RELAXED, __HIP_MEMORY_SCOPE_SYSTEM);
    } else if (lw == 1 && bid == 0) {
      GAS unsigned* d = (GAS unsigned*)(myws + WS_DONE);
      const u64 t0 = __builtin_amdgcn_s_memrealtime();
      bool bad = false;
      for (;;) {
        bool ok = true;
#pragma unroll
        for (int i = 0; i < 4; ++i) {
          const int e = lane + 64 * i;
          const unsigned v = __hip_atomic_load(d + e, __ATOMIC_RELAXED, __HIP_MEMORY_SCOPE_SYSTEM);
          ok &= ((e >> 5) == rank) || ((int)(v - call) >= 0);
        }
        const unsigned tv = __hip_atomic_load(tmo, __ATOMIC_RELAXED, __HIP_MEMORY_SCOPE_AGENT);
        if (__builtin_amdgcn_readfirstlane(tv) == call) { bad = true; break; }
        if (__all(ok)) break;
        if (__builtin_amdgcn_s_memrealtime() - t0 > DEADLINE_TICKS) {
          if (lane == 0) __hip_atomic_store(tmo, call, __ATOMIC_RELAXED, __HIP_MEMORY_SCOPE_AGENT);
          bad = true; break;
        }
        __builtin_amdgcn_s_sleep(1);
      }
      if (bad && lane == 0) p.out[rank][(size_t)rank * MLOC * NSH] = 0x7f;
    }
  } else {
    const int wm = wid >> 1, wn = wid & 1;
    const int fr = lane & 15, g = lane >> 4, q = (lane & 15) >> 2, pp = lane & 3;
    const unsigned xa0 = wm * 128 * 128 + fr * 128 + ((g ^ (fr >> 1)) * 16);
    unsigned wa[4];
#pragma unroll
    for (int ni = 0; ni < 4; ++ni)
      wa[ni] = W_BASE + (8 * g + q) * 256 + (((wn * 8 + 2 * pp + (ni >> 1)) ^ (8 * (g & 1) + q)) * 16) + (((ni ^ pp) & 1) * 8);
    f32x4 acc[8][4];
#pragma unroll
    for (int i = 0; i < 8; ++i)
#pragma unroll
      for (int j = 0; j < 4; ++j) acc[i][j] = (f32x4){0.f, 0.f, 0.f, 0.f};
    bf16x8 xA[8], wA[4], xB[8], wB[4];
    unsigned call = 0;
    if (wid == 0 && lane == 0)
      call = __hip_atomic_fetch_add((GAS unsigned*)(myws + WS_CNT + bid * 128), 1u, __ATOMIC_RELAXED, __HIP_MEMORY_SCOPE_AGENT) + 1u;
    call = __builtin_amdgcn_readfirstlane(call);
    BARRIER_L();
#pragma unroll
    for (int ni = 0; ni < 4; ++ni) wA[ni] = ldw(lds, wa[ni], 0);
#pragma unroll
    for (int mi = 0; mi < 8; ++mi) xA[mi] = ldx(lds, xa0, mi * 2048);
    unsigned xoff = 0, woff = 0;
    for (int t = 0; t < NT; ++t) {
      {
        const unsigned xa = (xa0 ^ 64) + xoff;
#pragma unroll
        for (int ni = 0; ni < 4; ++ni) wB[ni] = ldw(lds, wa[ni] + woff, 8192);
#pragma unroll
        for (int mi = 0; mi < 8; ++mi) xB[mi] = ldx(lds, xa, mi * 2048);
#pragma unroll
        for (int mi = 0; mi < 8; ++mi)
#pragma unroll
          for (int ni = 0; ni < 4; ++ni) acc[mi][ni] = __builtin_amdgcn_mfma_f32_16x16x32_bf16(wA[ni], xA[mi], acc[mi][ni], 0, 0, 0);
#pragma unroll
        for (int i = 0; i < 16; ++i) { __builtin_amdgcn_sched_group_barrier(0x8, 1, 0); __builtin_amdgcn_sched_group_barrier(0x100, 1, 0); }
        __builtin_amdgcn_sched_group_barrier(0x8, 16, 0);
      }
      __builtin_amdgcn_sched_barrier(0);
      BARRIER_L();
      __builtin_amdgcn_sched_barrier(0);
      xoff = (xoff == (XS - 1) * X_SLOT) ? 0 : xoff + X_SLOT; woff = (woff + W_SLOT) & (WS * W_SLOT - 1);
      {
        const unsigned xa = xa0 + xoff;
#pragma unroll
        for (int ni = 0; ni < 4; ++ni) wA[ni] = ldw(lds, wa[ni] + woff, 0);
#pragma unroll
        for (int mi = 0; mi < 8; ++mi) xA[mi] = ldx(lds, xa, mi * 2048);
#pragma unroll
        for (int mi = 0; mi < 8; ++mi)
#pragma unroll
          for (int ni = 0; ni < 4; ++ni) acc[mi][ni] = __builtin_amdgcn_mfma_f32_16x16x32_bf16(wB[ni], xB[mi], acc[mi][ni], 0, 0, 0);
#pragma unroll
        for (int i = 0; i < 16; ++i) { __builtin_amdgcn_sched_group_barrier(0x8, 1, 1); __builtin_amdgcn_sched_group_barrier(0x100, 1, 1); }
        __builtin_amdgcn_sched_group_barrier(0x8, 16, 1);
      }
      __builtin_amdgcn_sched_barrier(0);
    }
    float mx = 0.f;
#pragma unroll
    for (int mi = 0; mi < 8; ++mi)
#pragma unroll
      for (int ni = 0; ni < 4; ++ni)
#pragma unroll
        for (int r = 0; r < 4; ++r) { acc[mi][ni][r] = fmaxf(acc[mi][ni][r], 0.f); mx = fmaxf(mx, acc[mi][ni][r]); }
    mx = wave_max(mx);
    if (lane == 0) { red[wid] = mx; if (wid == 0) ((LAS unsigned*)red)[10] = call; }
    BARRIER_L();
    BARRIER_L();
    const float inv = red[8];
    const bool bad = red[9] != 0.f;
    const bool odd = (g & 1);
    unsigned char* obase = p.out[dst < (unsigned)NR ? dst : (unsigned)rank];
    unsigned char* orow = obase + (size_t)(rank * MLOC + mt * BM + wm * 128 + fr) * NSH + (panel & 15) * BN + wn * 64 + g * 16;
#pragma unroll
    for (int mi = 0; mi < 8; ++mi) {
      unsigned d[4];
#pragma unroll
      for (int ni = 0; ni < 4; ++ni) {
        const f32x4 v = acc[mi][ni] * inv;
        int r = __builtin_amdgcn_cvt_pk_fp8_f32(v[0], v[1], 0, false);
        r = __builtin_amdgcn_cvt_pk_fp8_f32(v[2], v[3], r, true);
        d[ni] = (unsigned)r;
      }
      u32x4 o;
      o[0] = odd ? d[1] : d[0]; o[1] = odd ? d[0] : d[1]; o[2] = odd ? d[3] : d[2]; o[3] = odd ? d[2] : d[3];
      if (bad) o = (u32x4){0x7f7f7f7fu, 0x7f7f7f7fu, 0x7f7f7f7fu, 0x7f7f7f7fu};
      asm volatile("global_store_dwordx4 %0, %1, off sc0 sc1\n\ts_nop 1" :: "v"(orow + (size_t)mi * 16 * NSH), "v"(o) : "memory");
    }
    BARRIER_V(0);
  }
}

}

extern "C" void kernel_launch(int rank, int nranks, ncclComm_t comm, hipStream_t stream,
                              void* const* d_in, const int* in_sizes, int n_in,
                              void* d_out, int out_size, const int* in_dtypes, int out_dtype,
                              void* const* d_out_all, void* const* d_ws_all, size_t ws_bytes) {
  (void)comm; (void)in_dtypes; (void)out_dtype; (void)d_out;
  if (nranks != NR || n_in != 2 || in_sizes[0] != MLOC * KDIM || in_sizes[1] != KDIM * NDIM ||
      out_size != NR * MLOC * NSH || ws_bytes < WS_END || (unsigned)rank >= (unsigned)NR) {
    fprintf(stderr, "unexpected problem shape\n");
    return;
  }
  Params p;
  p.x = (const unsigned short*)d_in[0];
  p.w = (const unsigned short*)d_in[1];
  for (int i = 0; i < NR; ++i) { p.out[i] = (unsigned char*)d_out_all[i]; p.ws[i] = (unsigned char*)d_ws_all[i]; }
  p.rank = rank; p.pad = 0;
  hipLaunchKernelGGL(fused_gemm_a2a, dim3(NWG), dim3(512), 0, stream, p);
  HIP_CHECK(hipGetLastError());
}
```

```cpp
#include <hip/hip_runtime.h>
#include <rccl/rccl.h>
#include <cstdio>

#define HIP_CHECK(expr) do { hipError_t _e = (expr); if (_e != hipSuccess) { \
    fprintf(stderr, "HIP error %s at %s:%d\n", hipGetErrorString(_e), __FILE__, __LINE__); \
    return; } } while (0)

namespace {

typedef short bf16x8 __attribute__((ext_vector_type(8)));
typedef short s16x4 __attribute__((ext_vector_type(4)));
typedef float f32x4 __attribute__((ext_vector_type(4)));
typedef unsigned u32x4 __attribute__((ext_vector_type(4)));
typedef unsigned long long u64;
#define LAS __attribute__((address_space(3)))
#define GAS __attribute__((address_space(1)))

constexpr int NR = 8;
constexpr int MLOC = 512, KDIM = 4096, NDIM = 16384, NSH = NDIM / NR;
constexpr int BM = 256, BN = 128, BK = 64, NT = KDIM / BK;
constexpr int NWG = (MLOC / BM) * (NDIM / BN);
constexpr int X_SLOT = BM * BK * 2, W_SLOT = BK * BN * 2, XS = 3, WS = 4;
constexpr int W_BASE = XS * X_SLOT, LDS_BYTES = W_BASE + WS * W_SLOT;

constexpr size_t WS_CNT = 0;
constexpr size_t WS_TMO = WS_CNT + NWG * 128;
constexpr size_t WS_AGG = WS_TMO + 256;
constexpr size_t WS_GLB = WS_AGG + 2 * NWG * 8;
constexpr size_t WS_DONE = WS_GLB + 2 * 128;
constexpr size_t WS_XFL = 64 * 1024;
constexpr size_t WS_XST = 1 << 20;
constexpr size_t WS_END = WS_XST + (size_t)NT * X_SLOT;
static_assert(WS_DONE + NR * 16 * 4 <= WS_XFL && WS_XFL + NR * 512 <= WS_XST, "workspace layout");
#define XWARM 1
#define LOOK 3
#define FENCE 1
#define PFX2 0
#define WNT 0
#define WEARLY 0
#define ROT 0
#define RAMP 1
#define NOPF 0

#define STAMP(i) do { } while (0)
#define TR(reg, id) do { } while (0)
#define TRDUMP(a, b) do { } while (0)
#define DEADL 50000000ull
constexpr u64 DEADLINE_TICKS = DEADL;

struct Params {
  const unsigned short* x;
  const unsigned short* w;
  unsigned char* out[NR];
  unsigned char* ws[NR];
  int rank;
  int pad;
};

#define BARRIER_L() asm volatile("s_waitcnt lgkmcnt(0)\n\ts_barrier" ::: "memory")
#define BARRIER_V(n) asm volatile("s_waitcnt vmcnt(" #n ")\n\ts_barrier" ::: "memory")

__device__ __forceinline__ bf16x8 ldx(LAS char* lds, unsigned addr, int imm) { return *(LAS bf16x8*)(lds + addr + imm); }
__device__ __forceinline__ bf16x8 ldw(LAS char* lds, unsigned addr, int imm) {
  s16x4 lo = __builtin_amdgcn_ds_read_tr16_b64_v4i16((LAS s16x4*)(lds + addr + imm));
  s16x4 hi = __builtin_amdgcn_ds_read_tr16_b64_v4i16((LAS s16x4*)(lds + addr + imm + 1024));
  return __builtin_shufflevector(lo, hi, 0, 1, 2, 3, 4, 5, 6, 7);
}
__device__ __forceinline__ float wave_max(float v) {
#pragma unroll
  for (int o = 32; o >= 1; o >>= 1) v = fmaxf(v, __shfl_xor(v, o));
  return v;
}
__device__ __forceinline__ unsigned wave_max_u(unsigned v) {
#pragma unroll
  for (int o = 32; o >= 1; o >>= 1) { unsigned t = (unsigned)__shfl_xor((int)v, o); v = v > t ? v : t; }
  return v;
}

__global__ __launch_bounds__(512, 2) void fused_gemm_a2a(const Params p) {
  __shared__ __attribute__((aligned(16))) char smem_[LDS_BYTES];
  LAS char* lds = (LAS char*)smem_;
  const int bid = blockIdx.x;
  const int xcd = bid & 7, idx = bid >> 3;
  const int rank = p.rank;
  const bool own = idx >= 16;
  const unsigned src = own ? (unsigned)rank : (unsigned)(rank + xcd) & (NR - 1);
  const int panel = own ? (idx - 16) * 8 + xcd : rank * 16 + idx;
  const int mt = own ? 0 : 1;
  const unsigned dst = (unsigned)(panel >> 4);
  const int wid = __builtin_amdgcn_readfirstlane(threadIdx.x >> 6);
  const int lane = threadIdx.x & 63;
  unsigned char* myws = p.ws[rank];
  LAS float* red = (LAS float*)lds;

  if (wid >= 4) {
    const int lw = wid - 4;
    GAS unsigned* tmo = (GAS unsigned*)(myws + WS_TMO);
    unsigned call;
    auto draw = [&](unsigned& c) {
      asm volatile("s_mov_b64 exec, 1\n\tglobal_atomic_add %0, %1, %2, off sc0\n\ts_mov_b64 exec, -1"
                   : "=v"(c) : "v"(myws + WS_CNT + bid * 128 + lw * 4), "v"(1u) : "memory");
    };
    auto drawn = [&](unsigned& c) {
      unsigned r;
      asm volatile("s_waitcnt vmcnt(0)\n\tv_readfirstlane_b32 %0, %1" : "=s"(r) : "v"(c) : "memory");
      return r + 1u;
    };
    unsigned char* lanews;
    auto pick_ws = [&]() {
      unsigned char* q[NR];
      asm volatile("" ::: "memory");
#pragma unroll
      for (int i = 0; i < NR; ++i) q[i] = p.ws[i];
      asm volatile("" : "+s"(q[0]), "+s"(q[1]), "+s"(q[2]), "+s"(q[3]), "+s"(q[4]), "+s"(q[5]), "+s"(q[6]), "+s"(q[7]));
      lanews = q[0];
#pragma unroll
      for (int i = 1; i < NR; ++i) lanews = lane == i ? q[i] : lanews;
      asm volatile("" : "+v"(lanews));
    };
    auto rw = __builtin_amdgcn_make_buffer_rsrc((void*)p.w, 0, KDIM * NDIM * 2, 0x00020000);
    const unsigned wv = (unsigned)((lane >> 4) * (NDIM * 2) + (((lane & 15) ^ ((lane >> 4) + 8 * (lw >> 1))) * 16) + panel * (BN * 2));
    unsigned trc = 0;
    if (own) {
      auto rx = __builtin_amdgcn_make_buffer_rsrc((void*)p.x, 0, BM * KDIM * 2, 0x00020000);
      const unsigned xv = (unsigned)((lane >> 3) * (KDIM * 2) + (((lane & 7) ^ (4 * (lw & 1) + (lane >> 4))) * 16));
      const unsigned xj0 = (lw >> 1) * 16 + (lw & 1);
      auto issue_x = [&](int t, int slot) {
        const unsigned so = (t < NT) ? (unsigned)(t * BK * 2) : 0x80000000u;
#pragma unroll
        for (int jj = 0; jj < 8; ++jj) {
          const unsigned j = xj0 + 2 * jj;
          __builtin_amdgcn_raw_ptr_buffer_load_lds(rx, (LAS void*)(lds + slot * X_SLOT + j * 1024), 16, xv, so + j * 8 * (KDIM * 2), 0, 0);
        }
      };
      auto issue_w = [&](int t, int slot) {
        const unsigned so = (t < NT) ? (unsigned)(t * BK * (NDIM * 2)) : 0x80000000u;
#pragma unroll
        for (int jj = 0; jj < 4; ++jj) {
          const unsigned j = lw + 4 * jj;
          __builtin_amdgcn_raw_ptr_buffer_load_lds(rw, (LAS void*)(lds + W_BASE + slot * W_SLOT + j * 1024), 16, wv, so + j * 4 * (NDIM * 2), 0, WNT);
        }
      };
      unsigned warm = 0, c;
      draw(c);
      if (XWARM) {
        const unsigned a = (unsigned)((bid - NWG / 2) * 16384 + lw * 4096 + lane * 64);
        asm volatile("buffer_load_dword %0, %1, %2, 0 offen" : "+v"(warm) : "v"(a), "s"(rx) : "memory");
      }
      issue_w(0, 0); issue_x(0, 0); issue_w(1, 1); issue_x(1, 1); issue_w(2, 2);
      pick_ws();
      int xs = 2, ws = 3;
      for (int t = 0; t < NT; ++t) {
        BARRIER_V(16);
        TR(trc, t);
        issue_x(t + 2, xs); issue_w(t + 3, ws);
        xs = (xs == XS - 1) ? 0 : xs + 1; ws = (ws + 1) & (WS - 1);
      }
      call = drawn(c);
      asm volatile("" : "+v"(warm));
    } else {
      unsigned c;
      draw(c);
      const int rot = ROT * xcd;
      auto issue_w = [&](int t, int slot) {
        const unsigned so = (t < NT) ? (unsigned)(((t + rot) & (NT - 1)) * BK * (NDIM * 2)) : 0x80000000u;
#pragma unroll
        for (int jj = 0; jj < 4; ++jj) {
          const unsigned j = lw + 4 * jj;
          __builtin_amdgcn_raw_ptr_buffer_load_lds(rw, (LAS void*)(lds + W_BASE + slot * W_SLOT + j * 1024), 16, wv, so + j * 4 * (NDIM * 2), 0, 0);
        }
      };
      const unsigned char* xb = p.ws[src] + WS_XST;
      auto rx = __builtin_amdgcn_make_buffer_rsrc((void*)xb, 0, NT * X_SLOT, 0x00020000);
      const unsigned xv = (unsigned)(lane * 16 + lw * 8192);
      auto issue_x = [&](int t, int slot) {
        const unsigned so = (t < NT) ? (unsigned)((t + rot) & (NT - 1)) * X_SLOT : 0x80000000u;
#pragma unroll
        for (int jj = 0; jj < 8; ++jj)
          __builtin_amdgcn_raw_ptr_buffer_load_lds(rx, (LAS void*)(lds + slot * X_SLOT + (8 * lw + jj) * 1024), 16, xv, so + jj * 1024, 0, 0);
      };
      const unsigned char* pb = xb + (idx * 4 + lw) * 512;
      u64 sink = 0;
      auto prefetch = [&](int t) {
        if (NOPF || t >= NT) return;
        const unsigned char* q = pb + ((t + rot) & (NT - 1)) * X_SLOT;
        if (PFX2) {
#pragma unroll
          for (int j = 0; j < 4; ++j) asm volatile("s_load_dwordx2 %0, %1, %2" : "+s"(sink) : "s"(q), "i"(j * 128 + 60) : "memory");
        } else {
          unsigned s32 = (unsigned)sink;
#pragma unroll
          for (int j = 0; j < 8; ++j) asm volatile("s_load_dword %0, %1, %2" : "+s"(s32) : "s"(q), "i"(j * 64) : "memory");
          sink = s32;
        }
      };
      pick_ws();
      if (FENCE == 0 && lw == 3) { __builtin_amdgcn_fence(__ATOMIC_ACQUIRE, ""); asm volatile("s_waitcnt vmcnt(0)" ::: "memory"); }
      BARRIER_L();
      call = drawn(c);
      const unsigned call0 = call;
      if (lw == 0) {
        if (lane < NR) __hip_atomic_store((GAS unsigned*)(lanews + WS_XFL) + rank * 128 + bid, call0, __ATOMIC_RELAXED, __HIP_MEMORY_SCOPE_SYSTEM);
        STAMP(7);
      }
      if (WEARLY) { issue_w(0, 0); issue_w(1, 1); issue_w(2, 2); }
      if (FENCE == 2 && lw == 1) { __builtin_amdgcn_fence(__ATOMIC_ACQUIRE, ""); asm volatile("s_waitcnt vmcnt(0)" ::: "memory"); }
      if (lw == 0) {
        const GAS u64* f = (const GAS u64*)(myws + WS_XFL + src * 512) + lane;
        const u64 t0 = __builtin_amdgcn_s_memrealtime();
        for (;;) {
          const u64 v = __hip_atomic_load(f, __ATOMIC_RELAXED, __HIP_MEMORY_SCOPE_SYSTEM);
          if (__all((int)((unsigned)v - call0) >= 0 && (int)((unsigned)(v >> 32) - call0) >= 0)) break;
          if (__builtin_amdgcn_readfirstlane(__hip_atomic_load(tmo, __ATOMIC_RELAXED, __HIP_MEMORY_SCOPE_AGENT)) == call0 ||
              __builtin_amdgcn_s_memrealtime() - t0 > DEADLINE_TICKS) {
            if (lane == 0) __hip_atomic_store(tmo, call0, __ATOMIC_RELAXED, __HIP_MEMORY_SCOPE_AGENT);
            break;
          }
          __builtin_amdgcn_s_sleep(1);
        }
        if (FENCE == 1) { __builtin_amdgcn_fence(__ATOMIC_ACQUIRE, ""); asm volatile("s_waitcnt vmcnt(0)" ::: "memory"); }
      }
      BARRIER_L();
      if (WEARLY) { issue_x(0, 0); issue_x(1, 1); } else { issue_x(0, 0); issue_w(0, 0); issue_x(1, 1); issue_w(1, 1); issue_w(2, 2); }
      int pf = 2;
      if (RAMP) { prefetch(pf++); } else { for (; pf < 2 + LOOK; ++pf) prefetch(pf); }
      if (WEARLY) BARRIER_V(8); else BARRIER_V(16);
      issue_x(2, 2); issue_w(3, 3); prefetch(pf++); if (pf < 3 + LOOK) prefetch(pf++);
      if (WEARLY) BARRIER_V(12); else BARRIER_V(16);
      int xs = 0, ws = 0;
      for (int t = 1; t < NT; ++t) {
        issue_x(t + 2, xs); issue_w(t + 3, ws); prefetch(pf++); if (pf < t + 3 + LOOK) prefetch(pf++);
        xs = (xs == XS - 1) ? 0 : xs + 1; ws = (ws + 1) & (WS - 1);
        if (t + 1 < NT) BARRIER_V(16);
      }
      asm volatile("s_waitcnt lgkmcnt(0)" : "+s"(sink) :: "memory");
    }
    BARRIER_V(0);
    BARRIER_L();
    const unsigned par = call & 1u;
    if (lw == 0) {
      const float m = fmaxf(fmaxf(red[0], red[1]), fmaxf(red[2], red[3]));
      if (lane == 0)
        __hip_atomic_store((GAS u64*)(myws + WS_AGG) + par * NWG + bid, ((u64)call << 32) | __float_as_uint(m), __ATOMIC_RELAXED, __HIP_MEMORY_SCOPE_AGENT);
      GAS u64* g = (GAS u64*)(myws + WS_GLB + par * 128);
      const u64 t0 = __builtin_amdgcn_s_memrealtime();
      unsigned mv = 0; bool bad = false;
      for (;;) {
        const u64 v = __hip_atomic_load(g + (lane & 7), __ATOMIC_RELAXED, __HIP_MEMORY_SCOPE_SYSTEM);
        const unsigned tv = __hip_atomic_load(tmo, __ATOMIC_RELAXED, __HIP_MEMORY_SCOPE_AGENT);
        mv = (unsigned)v;
        if (__builtin_amdgcn_readfirstlane(tv) == call) { bad = true; break; }
        if (__all((unsigned)(v >> 32) == call)) break;
        if (__builtin_amdgcn_s_memrealtime() - t0 > DEADLINE_TICKS) {
          if (lane == 0) __hip_atomic_store(tmo, call, __ATOMIC_RELAXED, __HIP_MEMORY_SCOPE_AGENT);
          bad = true; break;
        }
        __builtin_amdgcn_s_sleep(1);
      }
      unsigned am = mv;
#pragma unroll
      for (int o = 4; o >= 1; o >>= 1) { unsigned t = (unsigned)__shfl_xor((int)am, o); am = am > t ? am : t; }
      const float amax = __uint_as_float(am);
      const float scale = amax / 448.0f;
      float inv = amax > 0.f ? 1.0f / scale : 0.f;
      if (lane == 0) { red[8] = inv; red[9] = bad ? 1.f : 0.f; }
    } else if (lw == 1 && bid == 0) {
      GAS u64* a = (GAS u64*)(myws + WS_AGG) + par * NWG;
      const u64 t0 = __builtin_amdgcn_s_memrealtime();
      bool bad = false; unsigned mv = 0;
      for (;;) {
        u64 v[4];
#pragma unroll
        for (int i = 0; i < 4; ++i) v[i] = __hip_atomic_load(a + lane + 64 * i, __ATOMIC_RELAXED, __HIP_MEMORY_SCOPE_AGENT);
        bool ok = true; mv = 0;
#pragma unroll
        for (int i = 0; i < 4; ++i) { ok &= (unsigned)(v[i] >> 32) == call; mv = mv > (unsigned)v[i] ? mv : (unsigned)v[i]; }
        if (__all(ok)) break;
        if (__builtin_amdgcn_s_memrealtime() - t0 > DEADLINE_TICKS) {
          if (lane == 0) __hip_atomic_store(tmo, call, __ATOMIC_RELAXED, __HIP_MEMORY_SCOPE_AGENT);
          bad = true; break;
        }
      }
      STAMP(6);
      mv = wave_max_u(mv);
      if (!bad && lane < NR)
        __hip_atomic_store((GAS u64*)(lanews + WS_GLB + par * 128) + rank, ((u64)call << 32) | mv, __ATOMIC_RELAXED, __HIP_MEMORY_SCOPE_SYSTEM);
    }
    BARRIER_L();
    BARRIER_L();
    if (lw == 0) {
      if (lane == 0 && dst != (unsigned)rank && dst < (unsigned)NR)
        __hip_atomic_store((GAS unsigned*)(p.ws[dst] + WS_DONE) + rank * 16 + (panel & 15), call, __ATOMIC_RELAXED, __HIP_MEMORY_SCOPE_SYSTEM);
    } else if (lw == 1 && bid == 0) {
      GAS unsigned* d = (GAS unsigned*)(myws + WS_DONE);
      const u64 t0 = __builtin_amdgcn_s_memrealtime();
      bool bad = false;
      for (;;) {
        bool ok = true;
#pragma unroll
        for (int i = 0; i < 2; ++i) {
          const int e = lane + 64 * i;
          const unsigned v = __hip_atomic_load(d + e, __ATOMIC_RELAXED, __HIP_MEMORY_SCOPE_SYSTEM);
          ok &= ((e >> 4) == rank) || ((int)(v - call) >= 0);
        }
        const unsigned tv = __hip_atomic_load(tmo, __ATOMIC_RELAXED, __HIP_MEMORY_SCOPE_AGENT);
        if (__builtin_amdgcn_readfirstlane(tv) == call) { bad = true; break; }
        if (__all(ok)) break;
        if (__builtin_amdgcn_s_memrealtime() - t0 > DEADLINE_TICKS) {
          if (lane == 0) __hip_atomic_store(tmo, call, __ATOMIC_RELAXED, __HIP_MEMORY_SCOPE_AGENT);
          bad = true; break;
        }
        __builtin_amdgcn_s_sleep(1);
      }
      STAMP(5);
      if (bad && lane == 0) p.out[rank][(size_t)rank * MLOC * NSH] = 0x7f;
    }
  } else {
    if (wid == 0) STAMP(0);
    const int wm = wid >> 1, wn = wid & 1;
    const int fr = lane & 15, g = lane >> 4, q = (lane & 15) >> 2, pp = lane & 3;
    const unsigned xa0 = wm * 128 * 128 + fr * 128 + ((g ^ (fr >> 1)) * 16);
    unsigned wa[4];
#pragma unroll
    for (int ni = 0; ni < 4; ++ni)
      wa[ni] = W_BASE + (8 * g + q) * 256 + (((wn * 8 + 2 * pp + (ni >> 1)) ^ (8 * (g & 1) + q)) * 16) + (((ni ^ pp) & 1) * 8);
    f32x4 acc[8][4];
#pragma unroll
    for (int i = 0; i < 8; ++i)
#pragma unroll
      for (int j = 0; j < 4; ++j) acc[i][j] = (f32x4){0.f, 0.f, 0.f, 0.f};
    bf16x8 xA[8], wA[4], xB[8], wB[4];
    if (!own) {
      const int row = 2 * bid + (wid >> 1);
      const GAS char* sp = (const GAS char*)p.x + (size_t)(BM + row) * (KDIM * 2) + (wid & 1) * 4096 + lane * 16;
      unsigned char* dp = myws + WS_XST + (size_t)(32 * (wid & 1) + (lane >> 3)) * X_SLOT + row * 128 + (((lane & 7) ^ (bid & 7)) * 16);
      u32x4 c[4];
#pragma unroll
      for (int i = 0; i < 4; ++i) c[i] = *(const GAS u32x4*)(sp + i * 1024);
#pragma unroll
      for (int i = 0; i < 4; ++i)
        asm volatile("global_store_dwordx4 %0, %1, off sc0 sc1\n\ts_nop 1" :: "v"(dp + (size_t)i * 8 * X_SLOT), "v"(c[i]) : "memory");
      BARRIER_V(0);
      BARRIER_L();
    }
    BARRIER_L();
    if (wid == 0) STAMP(1);
#pragma unroll
    for (int ni = 0; ni < 4; ++ni) wA[ni] = ldw(lds, wa[ni], 0);
#pragma unroll
    for (int mi = 0; mi < 8; ++mi) xA[mi] = ldx(lds, xa0, mi * 2048);
    unsigned xoff = 0, woff = 0;
    for (int t = 0; t < NT; ++t) {
      {
        const unsigned xa = (xa0 ^ 64) + xoff;
#pragma unroll
        for (int ni = 0; ni < 4; ++ni) wB[ni] = ldw(lds, wa[ni] + woff, 8192);
#pragma unroll
        for (int mi = 0; mi < 8; ++mi) xB[mi] = ldx(lds, xa, mi * 2048);
#pragma unroll
        for (int mi = 0; mi < 8; ++mi)
#pragma unroll
          for (int ni = 0; ni < 4; ++ni) acc[mi][ni] = __builtin_amdgcn_mfma_f32_16x16x32_bf16(wA[ni], xA[mi], acc[mi][ni], 0, 0, 0);
#pragma unroll
        for (int i = 0; i < 16; ++i) { __builtin_amdgcn_sched_group_barrier(0x8, 1, 0); __builtin_amdgcn_sched_group_barrier(0x100, 1, 0); }
        __builtin_amdgcn_sched_group_barrier(0x8, 16, 0);
      }
      __builtin_amdgcn_sched_barrier(0);
      BARRIER_L();
      __builtin_amdgcn_sched_barrier(0);
      xoff = (xoff == (XS - 1) * X_SLOT) ? 0 : xoff + X_SLOT; woff = (woff + W_SLOT) & (WS * W_SLOT - 1);
      {
        const unsigned xa = xa0 + xoff;
#pragma unroll
        for (int ni = 0; ni < 4; ++ni) wA[ni] = ldw(lds, wa[ni] + woff, 0);
#pragma unroll
        for (int mi = 0; mi < 8; ++mi) xA[mi] = ldx(lds, xa, mi * 2048);
#pragma unroll
        for (int mi = 0; mi < 8; ++mi)
#pragma unroll
          for (int ni = 0; ni < 4; ++ni) acc[mi][ni] = __builtin_amdgcn_mfma_f32_16x16x32_bf16(wB[ni], xB[mi], acc[mi][ni], 0, 0, 0);
#pragma unroll
        for (int i = 0; i < 16; ++i) { __builtin_amdgcn_sched_group_barrier(0x8, 1, 1); __builtin_amdgcn_sched_group_barrier(0x100, 1, 1); }
        __builtin_amdgcn_sched_group_barrier(0x8, 16, 1);
      }
      __builtin_amdgcn_sched_barrier(0);
    }
    if (wid == 0) STAMP(2);
    float mx = 0.f;
#pragma unroll
    for (int mi = 0; mi < 8; ++mi)
#pragma unroll
      for (int ni = 0; ni < 4; ++ni)
#pragma unroll
        for (int r = 0; r < 4; ++r) { acc[mi][ni][r] = fmaxf(acc[mi][ni][r], 0.f); mx = fmaxf(mx, acc[mi][ni][r]); }
    mx = wave_max(mx);
    if (lane == 0) red[wid] = mx;
    BARRIER_L();
    BARRIER_L();
    if (wid == 0) STAMP(3);
    const float inv = red[8];
    const bool bad = red[9] != 0.f;
    const bool odd = (g & 1);
    unsigned char* obase = p.out[dst < (unsigned)NR ? dst : (unsigned)rank];
    unsigned char* orow = obase + (size_t)(src * MLOC + mt * BM + wm * 128 + fr) * NSH + (panel & 15) * BN + wn * 64 + g * 16;
#pragma unroll
    for (int mi = 0; mi < 8; ++mi) {
      unsigned d[4];
#pragma unroll
      for (int ni = 0; ni < 4; ++ni) {
        const f32x4 v = acc[mi][ni] * inv;
        int r = __builtin_amdgcn_cvt_pk_fp8_f32(v[0], v[1], 0, false);
        r = __builtin_amdgcn_cvt_pk_fp8_f32(v[2], v[3], r, true);
        d[ni] = (unsigned)r;
      }
      u32x4 o;
      o[0] = odd ? d[1] : d[0]; o[1] = odd ? d[0] : d[1]; o[2] = odd ? d[3] : d[2]; o[3] = odd ? d[2] : d[3];
      if (bad) o = (u32x4){0x7f7f7f7fu, 0x7f7f7f7fu, 0x7f7f7f7fu, 0x7f7f7f7fu};
      asm volatile("global_store_dwordx4 %0, %1, off sc0 sc1\n\ts_nop 1" :: "v"(orow + (size_t)mi * 16 * NSH), "v"(o) : "memory");
    }
    BARRIER_V(0);
    if (wid == 0) STAMP(4);
  }
}

}

extern "C" void kernel_launch(int rank, int nranks, ncclComm_t comm, hipStream_t stream,
                              void* const* d_in, const int* in_sizes, int n_in,
                              void* d_out, int out_size, const int* in_dtypes, int out_dtype,
                              void* const* d_out_all, void* const* d_ws_all, size_t ws_bytes) {
  (void)comm; (void)in_dtypes; (void)out_dtype; (void)d_out;
  if (nranks != NR || n_in != 2 || in_sizes[0] != MLOC * KDIM || in_sizes[1] != KDIM * NDIM ||
      out_size != NR * MLOC * NSH || ws_bytes < WS_END || (unsigned)rank >= (unsigned)NR) {
    fprintf(stderr, "unexpected problem shape\n");
    return;
  }
  Params p;
  p.x = (const unsigned short*)d_in[0];
  p.w = (const unsigned short*)d_in[1];
  for (int i = 0; i < NR; ++i) { p.out[i] = (unsigned char*)d_out_all[i]; p.ws[i] = (unsigned char*)d_ws_all[i]; }
  p.rank = rank; p.pad = 0;
  hipLaunchKernelGGL(fused_gemm_a2a, dim3(NWG), dim3(512), 0, stream, p);
  HIP_CHECK(hipGetLastError());
}
```

```cpp
#include <hip/hip_runtime.h>
#include <rccl/rccl.h>
#include <cstdio>

#define HIP_CHECK(expr) do { hipError_t _e = (expr); if (_e != hipSuccess) { \
    fprintf(stderr, "HIP error %s at %s:%d\n", hipGetErrorString(_e), __FILE__, __LINE__); \
    return; } } while (0)

namespace {

typedef short bf16x8 __attribute__((ext_vector_type(8)));
typedef short s16x4 __attribute__((ext_vector_type(4)));
typedef float f32x4 __attribute__((ext_vector_type(4)));
typedef unsigned u32x4 __attribute__((ext_vector_type(4)));
typedef unsigned u32x2 __attribute__((ext_vector_type(2)));
typedef unsigned long long u64;
#define LAS __attribute__((address_space(3)))
#define GAS __attribute__((address_space(1)))

constexpr int NR = 8;
constexpr int MLOC = 512, KDIM = 4096, NDIM = 16384, NSH = NDIM / NR;
constexpr int BM = 256, BN = 128, BK = 64, NT = KDIM / BK;
constexpr int NWG = (MLOC / BM) * (NDIM / BN);
constexpr int X_SLOT = BM * BK * 2, W_SLOT = BK * BN * 2, XS = 3, WS = 4;
constexpr int W_BASE = XS * X_SLOT, LDS_BYTES = W_BASE + WS * W_SLOT;
constexpr int L_STG = 32768, L_TBL = 4 * L_STG, L_SCALE = L_TBL + 256, L_FREE = L_SCALE + 16;

constexpr size_t WS_CNT = 0;
constexpr size_t WS_AGG = WS_CNT + NWG * 128;
constexpr size_t WS_GLB = WS_AGG + 2 * NWG * 4 * 8;
constexpr size_t WS_TKT = WS_GLB + 2 * 128;
constexpr size_t WS_FLG = WS_TKT + 64 * 128;
constexpr size_t WS_MSK = 1u << 20;
constexpr size_t WS_VAL = WS_MSK + (2u << 20);
constexpr size_t WS_END = WS_VAL + (16u << 20);
static_assert(WS_FLG + NR * 32 * 4 * 128 <= WS_MSK, "layout");

#define OST 1
#define PACE 68
#define TICKET 1
#define NAGG 4
#define STAMP(i) do { } while (0)
#define TS(v) do { } while (0)
#define TSOUT(i, v) do { } while (0)
constexpr u64 DEADLINE_TICKS = 50000000ull;

struct Params {
  const unsigned short* x;
  const unsigned short* w;
  unsigned char* out;
  unsigned char* ws[NR];
  int rank;
  int pad;
};

#define BARRIER_L() asm volatile("s_waitcnt lgkmcnt(0)\n\ts_barrier" ::: "memory")
#define BARRIER_V(n) asm volatile("s_waitcnt vmcnt(" #n ")\n\ts_barrier" ::: "memory")
#define WAIT_V0() asm volatile("s_waitcnt vmcnt(0)" ::: "memory")
#define STORE_SYS(ptr, val) asm volatile("global_store_dwordx4 %0, %1, off sc0 sc1\n\ts_nop 1" :: "v"(ptr), "v"(val) : "memory")
#define STORE_OUT(ptr, val) asm volatile("global_store_dwordx4 %0, %1, off sc0 sc1\n\ts_nop 1" :: "v"(ptr), "v"(val) : "memory")

__device__ __forceinline__ bf16x8 ldx(LAS char* lds, unsigned addr, int imm) { return *(LAS bf16x8*)(lds + addr + imm); }
__device__ __forceinline__ bf16x8 ldw(LAS char* lds, unsigned addr, int imm) {
  s16x4 lo = __builtin_amdgcn_ds_read_tr16_b64_v4i16((LAS s16x4*)(lds + addr + imm));
  s16x4 hi = __builtin_amdgcn_ds_read_tr16_b64_v4i16((LAS s16x4*)(lds + addr + imm + 1024));
  return __builtin_shufflevector(lo, hi, 0, 1, 2, 3, 4, 5, 6, 7);
}
#define DPP0(v, ctrl, rm) __builtin_amdgcn_update_dpp(0, (v), (ctrl), (rm), 0xf, false)
__device__ __forceinline__ int wave_max63(int v) {
  v = max(v, DPP0(v, 0x111, 0xf)); v = max(v, DPP0(v, 0x112, 0xf)); v = max(v, DPP0(v, 0x114, 0xf)); v = max(v, DPP0(v, 0x118, 0xf));
  v = max(v, DPP0(v, 0x142, 0xa)); v = max(v, DPP0(v, 0x143, 0xc));
  return v;
}
__device__ __forceinline__ int wave_scan(int v) {
  v += DPP0(v, 0x111, 0xf); v += DPP0(v, 0x112, 0xf); v += DPP0(v, 0x114, 0xf); v += DPP0(v, 0x118, 0xf);
  v += DPP0(v, 0x142, 0xa); v += DPP0(v, 0x143, 0xc);
  return v;
}
__device__ __forceinline__ void draw(unsigned& c, unsigned char* cnt) {
  asm volatile("s_mov_b64 exec, 1\n\tglobal_atomic_add %0, %1, %2, off sc0\n\ts_mov_b64 exec, -1" : "=v"(c) : "v"(cnt), "v"(1u) : "memory");
}
__device__ __forceinline__ unsigned drawn(unsigned& c) {
  unsigned r;
  asm volatile("s_waitcnt vmcnt(0)\n\tv_readfirstlane_b32 %0, %1" : "=s"(r) : "v"(c) : "memory");
  return r + 1u;
}
__device__ __forceinline__ unsigned pack4(f32x4 v) {
  int r = __builtin_amdgcn_cvt_pk_fp8_f32(v[0], v[1], 0, false);
  return (unsigned)__builtin_amdgcn_cvt_pk_fp8_f32(v[2], v[3], r, true);
}
__device__ __forceinline__ bool wait_flag(GAS unsigned* flag, unsigned want, u64 t0) {
  for (;;) {
    const unsigned v = __builtin_amdgcn_readfirstlane(__hip_atomic_load(flag, __ATOMIC_RELAXED, __HIP_MEMORY_SCOPE_SYSTEM));
    if ((int)(v - want) >= 0) return true;
    if (__builtin_amdgcn_s_memrealtime() - t0 > DEADLINE_TICKS) return false;
    __builtin_amdgcn_s_sleep(2);
  }
}

__global__ __launch_bounds__(512, 2) void fused_gemm_a2a(const Params p) {
  __shared__ __attribute__((aligned(16))) char smem_[LDS_BYTES];
  LAS char* lds = (LAS char*)smem_;
  const int bid = blockIdx.x;
  const int xcd = bid & 7, idx = bid >> 3;
  const int panel = (idx >> 1) * 8 + xcd, mt = idx & 1;
  const unsigned peer = (unsigned)(panel >> 4) & (NR - 1);
  const int q = (panel & 15) * 2 + mt;
  const int wid = __builtin_amdgcn_readfirstlane(threadIdx.x >> 6);
  const int lane = threadIdx.x & 63;
  const int rank = p.rank;
  const bool local = peer == (unsigned)rank;
  unsigned char* myws = p.ws[rank];
  const int fr = lane & 15, g = lane >> 4;
  const bool odd = g & 1;
  const int wv4 = wid & 3;
  unsigned char* orow = p.out + (size_t)(peer * MLOC + mt * BM + (wv4 >> 1) * 128 + fr) * NSH + (panel & 15) * BN + (wv4 & 1) * 64 + g * 16;
  unsigned cdraw;
  draw(cdraw, myws + WS_CNT + bid * 128 + wid * 4);

  if (wid >= 4) {
    const int lw = wid - 4;
    auto rx = __builtin_amdgcn_make_buffer_rsrc((void*)(p.x + (size_t)mt * BM * KDIM), 0, BM * KDIM * 2, 0x00020000);
    auto rw = __builtin_amdgcn_make_buffer_rsrc((void*)p.w, 0, KDIM * NDIM * 2, 0x00020000);
    const unsigned xv = (unsigned)((lane >> 3) * (KDIM * 2) + (((lane & 7) ^ (4 * (lw & 1) + (lane >> 4))) * 16));
    const unsigned xj0 = (lw >> 1) * 16 + (lw & 1);
    const unsigned wv = (unsigned)((lane >> 4) * (NDIM * 2) + (((lane & 15) ^ ((lane >> 4) + 8 * (lw >> 1))) * 16) + panel * (BN * 2));
    const unsigned wj0 = lw;
    auto issue_x = [&](int t, int slot) {
      const unsigned so = (t < NT) ? (unsigned)(t * BK * 2) : 0x80000000u;
#pragma unroll
      for (int jj = 0; jj < 8; ++jj) {
        const unsigned j = xj0 + 2 * jj;
        __builtin_amdgcn_raw_ptr_buffer_load_lds(rx, (LAS void*)(lds + slot * X_SLOT + j * 1024), 16, xv, so + j * 8 * (KDIM * 2), 0, 0);
      }
    };
    auto issue_w = [&](int t, int slot) {
      const unsigned so = (t < NT) ? (unsigned)(t * BK * (NDIM * 2)) : 0x80000000u;
#pragma unroll
      for (int jj = 0; jj < 4; ++jj) {
        const unsigned j = wj0 + 4 * jj;
        __builtin_amdgcn_raw_ptr_buffer_load_lds(rw, (LAS void*)(lds + W_BASE + slot * W_SLOT + j * 1024), 16, wv, so + j * 4 * (NDIM * 2), 0, 0);
      }
    };
    unsigned warm = 0;
    {
      auto rxa = __builtin_amdgcn_make_buffer_rsrc((void*)p.x, 0, MLOC * KDIM * 2, 0x00020000);
      const unsigned a = (unsigned)(bid * 16384 + lw * 4096 + lane * 64);
      asm volatile("buffer_load_dword %0, %1, %2, 0 offen" : "+v"(warm) : "v"(a), "s"(rxa) : "memory");
    }
    issue_w(0, 0); issue_x(0, 0); issue_w(1, 1); issue_x(1, 1); issue_w(2, 2);
    unsigned char* lanews;
    {
      unsigned char* qq[NR];
#pragma unroll
      for (int i = 0; i < NR; ++i) qq[i] = p.ws[i];
      asm volatile("" : "+s"(qq[0]), "+s"(qq[1]), "+s"(qq[2]), "+s"(qq[3]), "+s"(qq[4]), "+s"(qq[5]), "+s"(qq[6]), "+s"(qq[7]));
      lanews = qq[0];
#pragma unroll
      for (int i = 1; i < NR; ++i) lanews = lane == i ? qq[i] : lanews;
      asm volatile("" : "+v"(lanews));
    }
    unsigned tbl = 0;
    {
      unsigned pos = lane & 3;
#pragma unroll
      for (int k = 0; k < 4; ++k) {
        const unsigned keep = (lane >> (2 + k)) & 1;
        tbl |= (keep ? pos : 0x0cu) << (8 * k);
        pos += keep;
      }
      asm volatile("" : "+v"(tbl));
    }
    int xs = 2, ws = 3;
    for (int t = 0; t < NT; ++t) {
      BARRIER_V(16);
      issue_x(t + 2, xs); issue_w(t + 3, ws);
      xs = (xs == XS - 1) ? 0 : xs + 1; ws = (ws + 1) & (WS - 1);
    }
    const unsigned call = drawn(cdraw);
    asm volatile("" : "+v"(warm));
    if (lw == 0) *(LAS unsigned*)(lds + L_TBL + lane * 4) = tbl;
    if (lw == 1 && lane < 8) *(LAS unsigned*)(lds + L_SCALE + lane * 4) = 0u;
    BARRIER_L();
    const unsigned par = call & 1u;
    const u64 t0 = __builtin_amdgcn_s_memrealtime();
    if (local) {
      if (q == 0 && lw < NAGG) {
        unsigned char* a = myws + WS_AGG + par * (NWG * 4 * 8) + lane * 16;
        if (lw == 1) __builtin_amdgcn_s_sleep(8);
        if (lw == 2) __builtin_amdgcn_s_sleep(16);
        if (lw == 3) __builtin_amdgcn_s_sleep(24);
        bool ok; int mv;
        for (;;) {
          u32x4 v[8];
          asm volatile("global_load_dwordx4 %0, %8, off sc1\n\tglobal_load_dwordx4 %1, %8, off offset:1024 sc1\n\t"
                       "global_load_dwordx4 %2, %8, off offset:2048 sc1\n\tglobal_load_dwordx4 %3, %8, off offset:3072 sc1\n\t"
                       "global_load_dwordx4 %4, %9, off sc1\n\tglobal_load_dwordx4 %5, %9, off offset:1024 sc1\n\t"
                       "global_load_dwordx4 %6, %9, off offset:2048 sc1\n\tglobal_load_dwordx4 %7, %9, off offset:3072 sc1\n\ts_waitcnt vmcnt(0)"
                       : "=&v"(v[0]), "=&v"(v[1]), "=&v"(v[2]), "=&v"(v[3]), "=&v"(v[4]), "=&v"(v[5]), "=&v"(v[6]), "=&v"(v[7])
                       : "v"(a), "v"(a + 4096) : "memory");
          unsigned d = 0; mv = 0;
#pragma unroll
          for (int i = 0; i < 8; ++i) { d |= (v[i][1] ^ call) | (v[i][3] ^ call); mv = max(mv, max((int)v[i][0], (int)v[i][2])); }
          ok = __all(d == 0);
          if (ok || __builtin_amdgcn_s_memrealtime() - t0 > DEADLINE_TICKS) break;
        }
        if (lw == 0) STAMP(6);
        const unsigned m = (unsigned)__builtin_amdgcn_readlane(wave_max63(mv), 63);
        if (ok && lane < NR)
          __hip_atomic_store((GAS u64*)(lanews + WS_GLB + par * 128) + rank, ((u64)call << 32) | m, __ATOMIC_RELAXED, __HIP_MEMORY_SCOPE_SYSTEM);
      }
    } else {
      const unsigned slot = (peer * 32 + q) * 4 + lw;
      GAS unsigned* flag = (GAS unsigned*)(myws + WS_FLG + slot * 128);
      bool ok = wait_flag(flag, 2u * call, t0);
      TS(ts7);
      u32x4 mk;
      asm volatile("global_load_dwordx4 %0, %1, off sc0 sc1\n\ts_waitcnt vmcnt(0)"
                   : "=&v"(mk) : "v"(myws + WS_MSK + (size_t)(par * 1024 + slot) * 1024 + lane * 16) : "memory");
      const int cnt = __popc(mk[0]) + __popc(mk[1]) + __popc(mk[2]) + __popc(mk[3]);
      const int inc = wave_scan(cnt);
      const int n = __builtin_amdgcn_readlane(inc, 63);
      unsigned a = (unsigned)(inc - cnt) + lw * L_STG;
      unsigned ad[32], sel[32];
#pragma unroll
      for (int i = 0; i < 32; ++i) {
        const unsigned m4 = (mk[i >> 3] >> (4 * (i & 7))) & 15u;
        sel[i] = *(LAS unsigned*)(lds + L_TBL + ((m4 << 2 | (a & 3u)) << 2));
        ad[i] = a & ~3u;
        a += __popc(m4);
      }
#pragma unroll
      for (int i = 0; i < 32; i += 2) {
        const unsigned a0 = ad[i], a1 = ad[i + 1], s0 = sel[i], s1 = sel[i + 1];
        ad[i] = odd ? a1 : a0; ad[i + 1] = odd ? a0 : a1; sel[i] = odd ? s1 : s0; sel[i + 1] = odd ? s0 : s1;
      }
#pragma unroll
      for (int i = 0; i < 32; ++i) asm volatile("" : "+v"(ad[i]), "+v"(sel[i]));
      TS(ts8);
      while (ok && __hip_atomic_load((LAS unsigned*)(lds + L_FREE + lw * 4), __ATOMIC_ACQUIRE, __HIP_MEMORY_SCOPE_WORKGROUP) == 0u) {
        if (__builtin_amdgcn_s_memrealtime() - t0 > DEADLINE_TICKS) ok = false;
        __builtin_amdgcn_s_sleep(1);
      }
      if (ok) ok = wait_flag(flag, 2u * call + 1u, t0);
      TS(ts9);
      if (ok) {
        unsigned char* vs = myws + WS_VAL + (size_t)(par * 1024 + slot) * 8192 + lane * 16;
#pragma unroll
        for (int j = 0; j < 8; ++j)
          if (j * 1024 < n) __builtin_amdgcn_global_load_lds((GAS void*)(vs + j * 1024), (LAS void*)(lds + lw * L_STG + j * 1024), 16, 0, 17);
        WAIT_V0();
        TS(ts10);
        u32x2 d[32];
#pragma unroll
        for (int i = 0; i < 32; ++i) { d[i][0] = *(LAS unsigned*)(lds + ad[i]); d[i][1] = *(LAS unsigned*)(lds + ad[i] + 4); }
#pragma unroll
        for (int mi = 0; mi < 8; ++mi) {
          u32x4 o;
#pragma unroll
          for (int k = 0; k < 4; ++k) o[k] = __builtin_amdgcn_perm(d[mi * 4 + k][1], d[mi * 4 + k][0], sel[mi * 4 + k]);
          STORE_OUT(orow + (size_t)mi * 16 * NSH, o);
        }
        TS(ts11);
        TSOUT(16 + lw, ts10);
        if (lw == 0) TSOUT(9, ts11 - ts10);
      } else {
        const u32x4 o = {0x7f7f7f7fu, 0x7f7f7f7fu, 0x7f7f7f7fu, 0x7f7f7f7fu};
#pragma unroll
        for (int mi = 0; mi < 8; ++mi) STORE_OUT(orow + (size_t)mi * 16 * NSH, o);
      }
    }
  } else {
    if (wid == 0) STAMP(0);
    const int wm = wid >> 1, wn = wid & 1;
    const int qq = (lane & 15) >> 2, pp = lane & 3;
    const unsigned xa0 = wm * 128 * 128 + fr * 128 + ((g ^ (fr >> 1)) * 16);
    unsigned wa[4];
#pragma unroll
    for (int ni = 0; ni < 4; ++ni)
      wa[ni] = W_BASE + (8 * g + qq) * 256 + (((wn * 8 + 2 * pp + (ni >> 1)) ^ (8 * (g & 1) + qq)) * 16) + (((ni ^ pp) & 1) * 8);
    f32x4 acc[8][4];
#pragma unroll
    for (int i = 0; i < 8; ++i)
#pragma unroll
      for (int j = 0; j < 4; ++j) acc[i][j] = (f32x4){0.f, 0.f, 0.f, 0.f};
    bf16x8 xA[8], wA[4], xB[8], wB[4];
    unsigned char* peerws = p.ws[peer];
    asm volatile("" : "+s"(peerws));
    const unsigned call = drawn(cdraw);
    BARRIER_L();
    if (wid == 0) STAMP(1);
#pragma unroll
    for (int ni = 0; ni < 4; ++ni) wA[ni] = ldw(lds, wa[ni], 0);
#pragma unroll
    for (int mi = 0; mi < 8; ++mi) xA[mi] = ldx(lds, xa0, mi * 2048);
    unsigned xoff = 0, woff = 0;
    for (int t = 0; t < NT; ++t) {
      {
        const unsigned xa = (xa0 ^ 64) + xoff;
#pragma unroll
        for (int ni = 0; ni < 4; ++ni) wB[ni] = ldw(lds, wa[ni] + woff, 8192);
#pragma unroll
        for (int mi = 0; mi < 8; ++mi) xB[mi] = ldx(lds, xa, mi * 2048);
#pragma unroll
        for (int mi = 0; mi < 8; ++mi)
#pragma unroll
          for (int ni = 0; ni < 4; ++ni) acc[mi][ni] = __builtin_amdgcn_mfma_f32_16x16x32_bf16(wA[ni], xA[mi], acc[mi][ni], 0, 0, 0);
#pragma unroll
        for (int i = 0; i < 16; ++i) { __builtin_amdgcn_sched_group_barrier(0x8, 1, 0); __builtin_amdgcn_sched_group_barrier(0x100, 1, 0); }
        __builtin_amdgcn_sched_group_barrier(0x8, 16, 0);
      }
      __builtin_amdgcn_sched_barrier(0);
      BARRIER_L();
      __builtin_amdgcn_sched_barrier(0);
      xoff = (xoff == (XS - 1) * X_SLOT) ? 0 : xoff + X_SLOT; woff = (woff + W_SLOT) & (WS * W_SLOT - 1);
      {
        const unsigned xa = xa0 + xoff;
#pragma unroll
        for (int ni = 0; ni < 4; ++ni) wA[ni] = ldw(lds, wa[ni] + woff, 0);
#pragma unroll
        for (int mi = 0; mi < 8; ++mi) xA[mi] = ldx(lds, xa, mi * 2048);
#pragma unroll
        for (int mi = 0; mi < 8; ++mi)
#pragma unroll
          for (int ni = 0; ni < 4; ++ni) acc[mi][ni] = __builtin_amdgcn_mfma_f32_16x16x32_bf16(wB[ni], xB[mi], acc[mi][ni], 0, 0, 0);
#pragma unroll
        for (int i = 0; i < 16; ++i) { __builtin_amdgcn_sched_group_barrier(0x8, 1, 1); __builtin_amdgcn_sched_group_barrier(0x100, 1, 1); }
        __builtin_amdgcn_sched_group_barrier(0x8, 16, 1);
      }
      __builtin_amdgcn_sched_barrier(0);
    }
    if (wid == 0) STAMP(2);
    const unsigned par = call & 1u;
    {
      int mx = 0;
#pragma unroll
      for (int mi = 0; mi < 8; ++mi)
#pragma unroll
        for (int ni = 0; ni < 4; ++ni) {
          mx = max(max(mx, __float_as_int(acc[mi][ni][0])), __float_as_int(acc[mi][ni][1]));
          mx = max(max(mx, __float_as_int(acc[mi][ni][2])), __float_as_int(acc[mi][ni][3]));
        }
      mx = wave_max63(mx);
      if (lane == 63)
        __hip_atomic_store((GAS u64*)(myws + WS_AGG) + par * (NWG * 4) + bid * 4 + wid, ((u64)call << 32) | (unsigned)mx, __ATOMIC_RELAXED, __HIP_MEMORY_SCOPE_AGENT);
    }
    __builtin_amdgcn_sched_barrier(0);
    const u64 t0 = __builtin_amdgcn_s_memrealtime();
    const unsigned slot = (rank * 32 + q) * 4 + wid;
    GAS unsigned* flag = (GAS unsigned*)(peerws + WS_FLG + slot * 128);
    int n = 0;
    unsigned tk = 0;
    if (!local) {
      if (PACE && TICKET) draw(tk, myws + WS_TKT + (xcd * 8 + peer) * 128);
      acc[7][3][3] = __int_as_float(max(__float_as_int(acc[7][3][3]), 0));
      u32x4 mk;
#pragma unroll
      for (int w = 0; w < 4; ++w) {
        unsigned m = 0;
#pragma unroll
        for (int i = 31; i >= 0; --i) m = __builtin_amdgcn_alignbit(m, __float_as_uint(acc[2 * w + (i >> 4)][(i >> 2) & 3][i & 3]), 31);
        mk[w] = ~m;
      }
      TS(tsa);
      STORE_SYS(peerws + WS_MSK + (size_t)(par * 1024 + slot) * 1024 + lane * 16, mk);
      const int cnt = __popc(mk[0]) + __popc(mk[1]) + __popc(mk[2]) + __popc(mk[3]);
      const int inc = wave_scan(cnt);
      n = __builtin_amdgcn_readlane(inc, 63);
      unsigned a = (unsigned)(size_t)lds + wid * L_STG + (inc - cnt) * 4;
#pragma unroll
      for (int i = 0; i < 32; ++i) {
        const f32x4 v = acc[i >> 2][i & 3];
        int sg;
        asm volatile("ds_write_b32 %0, %2 offset:%6\n\tv_ashrrev_i32 %1, 31, %2\n\tv_lshl_add_u32 %0, %1, 2, %0\n\t"
                     "ds_write_b32 %0, %3 offset:%6+4\n\tv_ashrrev_i32 %1, 31, %3\n\tv_lshl_add_u32 %0, %1, 2, %0\n\t"
                     "ds_write_b32 %0, %4 offset:%6+8\n\tv_ashrrev_i32 %1, 31, %4\n\tv_lshl_add_u32 %0, %1, 2, %0\n\t"
                     "ds_write_b32 %0, %5 offset:%6+12\n\tv_ashrrev_i32 %1, 31, %5\n\tv_lshl_add_u32 %0, %1, 2, %0"
                     : "+v"(a), "=&v"(sg) : "v"(v[0]), "v"(v[1]), "v"(v[2]), "v"(v[3]), "n"(16 * i) : "memory");
      }
      TS(tsc);
      if (PACE && TICKET) tk = (15u - ((drawn(tk) - 1u) & 15u)) * 8 + xcd; else tk = ((idx & 3) * 4 + wid) * 8 + xcd;
      WAIT_V0();
      TS(tsd);
      if (lane == 0) __hip_atomic_store(flag, 2u * call, __ATOMIC_RELAXED, __HIP_MEMORY_SCOPE_SYSTEM);
#pragma unroll
      for (int j = 0; j < 8; ++j)
        if (j * 1024 < n) {
#pragma unroll
          for (int k = 0; k < 4; ++k) acc[j][k] = *(LAS f32x4*)(lds + wid * L_STG + j * 4096 + lane * 64 + k * 16);
        }
      asm volatile("s_waitcnt lgkmcnt(0)" ::: "memory");
      if (lane == 0) __hip_atomic_store((LAS unsigned*)(lds + L_FREE + wid * 4), 1u, __ATOMIC_RELEASE, __HIP_MEMORY_SCOPE_WORKGROUP);
    }
    if (wid == 0) STAMP(5);
    float inv; bool bad;
    if (wid == 0) {
      GAS u64* gl = (GAS u64*)(myws + WS_GLB + par * 128);
      int mv;
      for (;;) {
        const u64 v = __hip_atomic_load(gl + (lane & 7), __ATOMIC_RELAXED, __HIP_MEMORY_SCOPE_SYSTEM);
        mv = (int)(unsigned)v;
        bad = !__all((unsigned)(v >> 32) == call);
        if (!bad || __builtin_amdgcn_s_memrealtime() - t0 > DEADLINE_TICKS) break;
        __builtin_amdgcn_s_sleep(1);
      }
      mv = max(mv, DPP0(mv, 0x111, 0xf)); mv = max(mv, DPP0(mv, 0x112, 0xf)); mv = max(mv, DPP0(mv, 0x114, 0xf));
      const float amax = __int_as_float(__builtin_amdgcn_readlane(mv, 7));
      const float scale = amax / 448.0f;
      inv = amax > 0.f ? 1.0f / scale : 0.f;
      if (lane == 0)
        __hip_atomic_store((LAS u64*)(lds + L_SCALE), ((u64)(bad ? 2u : 1u) << 32) | __float_as_uint(inv), __ATOMIC_RELEASE, __HIP_MEMORY_SCOPE_WORKGROUP);
    } else {
      u64 v;
      for (;;) {
        v = __hip_atomic_load((LAS u64*)(lds + L_SCALE), __ATOMIC_ACQUIRE, __HIP_MEMORY_SCOPE_WORKGROUP);
        if (__builtin_amdgcn_readfirstlane((unsigned)(v >> 32)) != 0u) break;
        if (__builtin_amdgcn_s_memrealtime() - t0 > 2 * DEADLINE_TICKS) { v = 2ull << 32; break; }
        __builtin_amdgcn_s_sleep(1);
      }
      inv = __uint_as_float((unsigned)v);
      bad = __builtin_amdgcn_readfirstlane((unsigned)(v >> 32)) != 1u;
    }
    if (wid == 0) STAMP(3);
    const u32x4 nan4 = {0x7f7f7f7fu, 0x7f7f7f7fu, 0x7f7f7f7fu, 0x7f7f7f7fu};
    if (local) {
#pragma unroll
      for (int mi = 0; mi < 8; ++mi) {
        unsigned d[4];
#pragma unroll
        for (int ni = 0; ni < 4; ++ni) {
          f32x4 v;
#pragma unroll
          for (int r = 0; r < 4; ++r) v[r] = __int_as_float(max(__float_as_int(acc[mi][ni][r]), 0));
          d[ni] = pack4(v * inv);
        }
        u32x4 o;
        o[0] = odd ? d[1] : d[0]; o[1] = odd ? d[0] : d[1]; o[2] = odd ? d[3] : d[2]; o[3] = odd ? d[2] : d[3];
        if (bad) o = nan4;
        STORE_OUT(orow + (size_t)mi * 16 * NSH, o);
      }
    } else {
      if (PACE) {
        const unsigned ts = (unsigned)__builtin_amdgcn_s_memrealtime(), dt = (tk * PACE * (unsigned)n) / 40960u;
        while ((unsigned)__builtin_amdgcn_s_memrealtime() - ts < dt) __builtin_amdgcn_s_sleep(1);
      }
      unsigned char* vd = peerws + WS_VAL + (size_t)(par * 1024 + slot) * 8192 + lane * 16;
#pragma unroll
      for (int j = 0; j < 8; ++j)
        if (j * 1024 < n) {
          u32x4 o;
#pragma unroll
          for (int k = 0; k < 4; ++k) o[k] = pack4(acc[j][k] * inv);
          if (bad) o = nan4;
          if (j * 1024 + lane * 16 < n) STORE_SYS(vd + j * 1024, o);
        }
      WAIT_V0();
      if (lane == 0) __hip_atomic_store(flag, 2u * call + 1u, __ATOMIC_RELAXED, __HIP_MEMORY_SCOPE_SYSTEM);
    }
  }
}

}

extern "C" void kernel_launch(int rank, int nranks, ncclComm_t comm, hipStream_t stream,
                              void* const* d_in, const int* in_sizes, int n_in,
                              void* d_out, int out_size, const int* in_dtypes, int out_dtype,
                              void* const* d_out_all, void* const* d_ws_all, size_t ws_bytes) {
  (void)comm; (void)in_dtypes; (void)out_dtype; (void)d_out_all;
  if (nranks != NR || n_in != 2 || in_sizes[0] != MLOC * KDIM || in_sizes[1] != KDIM * NDIM ||
      out_size != NR * MLOC * NSH || ws_bytes < WS_END || (unsigned)rank >= (unsigned)NR) {
    fprintf(stderr, "unexpected problem shape\n");
    return;
  }
  Params p;
  p.x = (const unsigned short*)d_in[0];
  p.w = (const unsigned short*)d_in[1];
  p.out = (unsigned char*)d_out;
  for (int i = 0; i < NR; ++i) p.ws[i] = (unsigned char*)d_ws_all[i];
  p.rank = rank; p.pad = 0;
  hipLaunchKernelGGL(fused_gemm_a2a, dim3(NWG), dim3(512), 0, stream, p);
  HIP_CHECK(hipGetLastError());
}
```

```cpp
#include <hip/hip_runtime.h>
#include <rccl/rccl.h>
#include <cstdio>

#define HIP_CHECK(expr) do { hipError_t _e = (expr); if (_e != hipSuccess) { \
    fprintf(stderr, "HIP error %s at %s:%d\n", hipGetErrorString(_e), __FILE__, __LINE__); \
    return; } } while (0)

namespace {

typedef short bf16x8 __attribute__((ext_vector_type(8)));
typedef short s16x4 __attribute__((ext_vector_type(4)));
typedef float f32x4 __attribute__((ext_vector_type(4)));
typedef float f32x2 __attribute__((ext_vector_type(2)));
typedef unsigned u32x4 __attribute__((ext_vector_type(4)));
typedef unsigned u32x2 __attribute__((ext_vector_type(2)));
typedef unsigned long long u64;
#define LAS __attribute__((address_space(3)))
#define GAS __attribute__((address_space(1)))

constexpr int NR = 8;
constexpr int MLOC = 512, KDIM = 4096, NDIM = 16384, NSH = NDIM / NR;
constexpr int BM = 256, BN = 128, BK = 64, NT = KDIM / BK;
constexpr int NWG = (MLOC / BM) * (NDIM / BN);
constexpr int X_SLOT = BM * BK * 2, W_SLOT = BK * BN * 2, XS = 3, WS = 4;
constexpr int W_BASE = XS * X_SLOT, LDS_BYTES = W_BASE + WS * W_SLOT;
constexpr int L_STG = 32768, L_TBL = 4 * L_STG, L_SCALE = L_TBL + 256, L_FREE = L_SCALE + 16;

constexpr size_t WS_CNT = 0;
constexpr size_t WS_AGG = WS_CNT + NWG * 128;
constexpr size_t WS_GLB = WS_AGG + 2 * NWG * 4 * 8;
constexpr size_t WS_TKT = WS_GLB + 2 * 128;
constexpr size_t WS_FLG = WS_TKT + 64 * 128;
constexpr size_t WS_MSK = 1u << 20;
constexpr size_t WS_VAL = WS_MSK + (2u << 20);
constexpr size_t WS_END = WS_VAL + (16u << 20);
static_assert(WS_FLG + NR * 32 * 4 * 128 <= WS_MSK, "layout");

#define OST 1
#define PACE 70
#define GATE 40
#define CMPV 3
#define DPOS 0
#define WPOS 0
#define NAGG 4
#define STAMP(i) do { } while (0)
#define TS(v) do { } while (0)
#define TSOUT(i, v) do { } while (0)
constexpr u64 DEADLINE_TICKS = 50000000ull;

struct Params {
  const unsigned short* x;
  const unsigned short* w;
  unsigned char* out;
  unsigned char* ws[NR];
  int rank;
  int pad;
};

#define BARRIER_L() asm volatile("s_waitcnt lgkmcnt(0)\n\ts_barrier" ::: "memory")
#define BARRIER_V(n) asm volatile("s_waitcnt vmcnt(" #n ")\n\ts_barrier" ::: "memory")
#define WAIT_V0() asm volatile("s_waitcnt vmcnt(0)" ::: "memory")
#define STORE_SYS(ptr, val) asm volatile("global_store_dwordx4 %0, %1, off sc0 sc1\n\ts_nop 1" :: "v"(ptr), "v"(val) : "memory")
#define OBITS " sc0 sc1"
#define STORE_OUT(mi, val) asm volatile("buffer_store_dwordx4 %0, %1, %2, %3 offen" OBITS "\n\ts_nop 1" :: "v"(val), "v"(ovo), "s"(rout), "s"((mi) * 16 * NSH) : "memory")

__device__ __forceinline__ bf16x8 ldx(LAS char* lds, unsigned addr, int imm) { return *(LAS bf16x8*)(lds + addr + imm); }
__device__ __forceinline__ bf16x8 ldw(LAS char* lds, unsigned addr, int imm) {
  s16x4 lo = __builtin_amdgcn_ds_read_tr16_b64_v4i16((LAS s16x4*)(lds + addr + imm));
  s16x4 hi = __builtin_amdgcn_ds_read_tr16_b64_v4i16((LAS s16x4*)(lds + addr + imm + 1024));
  return __builtin_shufflevector(lo, hi, 0, 1, 2, 3, 4, 5, 6, 7);
}
#define DPP0(v, ctrl, rm) __builtin_amdgcn_update_dpp(0, (v), (ctrl), (rm), 0xf, false)
__device__ __forceinline__ int wave_max63(int v) {
  v = max(v, DPP0(v, 0x111, 0xf)); v = max(v, DPP0(v, 0x112, 0xf)); v = max(v, DPP0(v, 0x114, 0xf)); v = max(v, DPP0(v, 0x118, 0xf));
  v = max(v, DPP0(v, 0x142, 0xa)); v = max(v, DPP0(v, 0x143, 0xc));
  return v;
}
__device__ __forceinline__ int wave_scan(int v) {
  v += DPP0(v, 0x111, 0xf); v += DPP0(v, 0x112, 0xf); v += DPP0(v, 0x114, 0xf); v += DPP0(v, 0x118, 0xf);
  v += DPP0(v, 0x142, 0xa); v += DPP0(v, 0x143, 0xc);
  return v;
}
__device__ __forceinline__ void draw(unsigned& c, unsigned char* cnt) {
  asm volatile("s_mov_b64 exec, 1\n\tglobal_atomic_add %0, %1, %2, off sc0\n\ts_mov_b64 exec, -1" : "=v"(c) : "v"(cnt), "v"(1u) : "memory");
}
__device__ __forceinline__ unsigned drawn(unsigned& c) {
  unsigned r;
  asm volatile("s_waitcnt vmcnt(0)\n\tv_readfirstlane_b32 %0, %1" : "=s"(r) : "v"(c) : "memory");
  return r + 1u;
}
__device__ __forceinline__ unsigned pack4(f32x4 v) {
  int r = __builtin_amdgcn_cvt_pk_fp8_f32(v[0], v[1], __float_as_int(v[0]), false);
  return (unsigned)__builtin_amdgcn_cvt_pk_fp8_f32(v[2], v[3], r, true);
}
__device__ __forceinline__ unsigned bfi(unsigned m, unsigned a, unsigned b) { return (m & a) | (~m & b); }

__global__ __launch_bounds__(512, 2) void fused_gemm_a2a(const Params p) {
  __shared__ __attribute__((aligned(16))) char smem_[LDS_BYTES];
  LAS char* lds = (LAS char*)smem_;
  const int bid = blockIdx.x;
  const int xcd = bid & 7, idx = bid >> 3;
  const int panel = (idx >> 1) * 8 + xcd, mt = idx & 1;
  const unsigned peer = (unsigned)(panel >> 4) & (NR - 1);
  const int q = (panel & 15) * 2 + mt;
  const int wid = __builtin_amdgcn_readfirstlane(threadIdx.x >> 6);
  const int lane = threadIdx.x & 63;
  const int rank = p.rank;
  const bool local = peer == (unsigned)rank;
  unsigned char* myws = p.ws[rank];
  const int fr = lane & 15, g = lane >> 4;
  const bool odd = g & 1;
  const int wv4 = wid & 3;
  const unsigned ovo = (unsigned)((peer * MLOC + mt * BM + (wv4 >> 1) * 128 + fr) * NSH + (panel & 15) * BN + (wv4 & 1) * 64 + g * 16);
  const auto rout = __builtin_amdgcn_make_buffer_rsrc((void*)p.out, 0, NR * MLOC * NSH, 0x00020000);
  unsigned cdraw;
  if (DPOS == 0 || wid < 4) draw(cdraw, myws + WS_CNT + bid * 128 + wid * 4);

  if (wid >= 4) {
    const int lw = wid - 4;
    auto rx = __builtin_amdgcn_make_buffer_rsrc((void*)(p.x + (size_t)mt * BM * KDIM), 0, BM * KDIM * 2, 0x00020000);
    auto rw = __builtin_amdgcn_make_buffer_rsrc((void*)p.w, 0, KDIM * NDIM * 2, 0x00020000);
    const unsigned xv = (unsigned)((lane >> 3) * (KDIM * 2) + (((lane & 7) ^ (4 * (lw & 1) + (lane >> 4))) * 16));
    const unsigned xj0 = (lw >> 1) * 16 + (lw & 1);
    const unsigned wv = (unsigned)((lane >> 4) * (NDIM * 2) + (((lane & 15) ^ ((lane >> 4) + 8 * (lw >> 1))) * 16) + panel * (BN * 2));
    const unsigned wj0 = lw;
    auto issue_x = [&](int t, int slot) {
      const unsigned so = (t < NT) ? (unsigned)(t * BK * 2) : 0x80000000u;
#pragma unroll
      for (int jj = 0; jj < 8; ++jj) {
        const unsigned j = xj0 + 2 * jj;
        __builtin_amdgcn_raw_ptr_buffer_load_lds(rx, (LAS void*)(lds + slot * X_SLOT + j * 1024), 16, xv, so + j * 8 * (KDIM * 2), 0, 0);
      }
    };
    auto issue_w = [&](int t, int slot) {
      const unsigned so = (t < NT) ? (unsigned)(t * BK * (NDIM * 2)) : 0x80000000u;
#pragma unroll
      for (int jj = 0; jj < 4; ++jj) {
        const unsigned j = wj0 + 4 * jj;
        __builtin_amdgcn_raw_ptr_buffer_load_lds(rw, (LAS void*)(lds + W_BASE + slot * W_SLOT + j * 1024), 16, wv, so + j * 4 * (NDIM * 2), 0, 0);
      }
    };
    unsigned warm = 0;
    auto warmx = [&]() {
      auto rxa = __builtin_amdgcn_make_buffer_rsrc((void*)p.x, 0, MLOC * KDIM * 2, 0x00020000);
      const unsigned a = (unsigned)(bid * 16384 + lw * 4096 + lane * 64);
      asm volatile("buffer_load_dword %0, %1, %2, 0 offen" : "+v"(warm) : "v"(a), "s"(rxa) : "memory");
    };
    if (WPOS == 0) warmx();
    issue_w(0, 0); issue_x(0, 0);
    if (WPOS == 1) warmx();
    issue_w(1, 1); issue_x(1, 1); issue_w(2, 2);
    if (WPOS == 2) warmx();
    if (DPOS == 1) draw(cdraw, myws + WS_CNT + bid * 128 + wid * 4);
    unsigned char* lanews;
    {
      unsigned char* qq[NR];
#pragma unroll
      for (int i = 0; i < NR; ++i) qq[i] = p.ws[i];
      asm volatile("" : "+s"(qq[0]), "+s"(qq[1]), "+s"(qq[2]), "+s"(qq[3]), "+s"(qq[4]), "+s"(qq[5]), "+s"(qq[6]), "+s"(qq[7]));
      lanews = qq[0];
#pragma unroll
      for (int i = 1; i < NR; ++i) lanews = lane == i ? qq[i] : lanews;
      asm volatile("" : "+v"(lanews));
    }
    unsigned tbl = 0;
    {
      unsigned pos = lane & 3;
#pragma unroll
      for (int k = 0; k < 4; ++k) {
        const unsigned keep = (lane >> (2 + k)) & 1;
        tbl |= (keep ? pos : 0x0cu) << (8 * k);
        pos += keep;
      }
      asm volatile("" : "+v"(tbl));
    }
    int xs = 2, ws = 3;
    for (int t = 0; t < NT; ++t) {
      BARRIER_V(16);
      issue_x(t + 2, xs); issue_w(t + 3, ws);
      xs = (xs == XS - 1) ? 0 : xs + 1; ws = (ws + 1) & (WS - 1);
    }
    const unsigned call = drawn(cdraw);
    asm volatile("" : "+v"(warm));
    if (lw == 0) *(LAS unsigned*)(lds + L_TBL + lane * 4) = tbl;
    if (lw == 1 && lane < 8) *(LAS unsigned*)(lds + L_SCALE + lane * 4) = 0u;
    BARRIER_L();
    const unsigned par = call & 1u;
    const u64 t0 = __builtin_amdgcn_s_memrealtime();
    if (local) {
      if (q == 0 && lw < NAGG) {
        unsigned char* a = myws + WS_AGG + par * (NWG * 4 * 8) + lane * 16;
        if (lw == 1) __builtin_amdgcn_s_sleep(8);
        if (lw == 2) __builtin_amdgcn_s_sleep(16);
        if (lw == 3) __builtin_amdgcn_s_sleep(24);
        bool ok; int mv;
        for (;;) {
          u32x4 v[8];
          asm volatile("global_load_dwordx4 %0, %8, off sc1\n\tglobal_load_dwordx4 %1, %8, off offset:1024 sc1\n\t"
                       "global_load_dwordx4 %2, %8, off offset:2048 sc1\n\tglobal_load_dwordx4 %3, %8, off offset:3072 sc1\n\t"
                       "global_load_dwordx4 %4, %9, off sc1\n\tglobal_load_dwordx4 %5, %9, off offset:1024 sc1\n\t"
                       "global_load_dwordx4 %6, %9, off offset:2048 sc1\n\tglobal_load_dwordx4 %7, %9, off offset:3072 sc1\n\ts_waitcnt vmcnt(0)"
                       : "=&v"(v[0]), "=&v"(v[1]), "=&v"(v[2]), "=&v"(v[3]), "=&v"(v[4]), "=&v"(v[5]), "=&v"(v[6]), "=&v"(v[7])
                       : "v"(a), "v"(a + 4096) : "memory");
          unsigned d = 0; mv = 0;
#pragma unroll
          for (int i = 0; i < 8; ++i) { d |= (v[i][1] ^ call) | (v[i][3] ^ call); mv = max(mv, max((int)v[i][0], (int)v[i][2])); }
          ok = __all(d == 0);
          if (ok || __builtin_amdgcn_s_memrealtime() - t0 > DEADLINE_TICKS) break;
        }
        if (lw == 0) STAMP(6);
        const unsigned m = (unsigned)__builtin_amdgcn_readlane(wave_max63(mv), 63);
        if (ok && lane < NR)
          __hip_atomic_store((GAS u64*)(lanews + WS_GLB + par * 128) + rank, ((u64)call << 32) | m, __ATOMIC_RELAXED, __HIP_MEMORY_SCOPE_SYSTEM);
      }
    } else {
      const unsigned slot = (peer * 32 + q) * 4 + lw;
      GAS u64* flag = (GAS u64*)(myws + WS_FLG + slot * 128);
      bool ok; unsigned tk;
      for (;;) {
        const u64 v = __hip_atomic_load(flag, __ATOMIC_RELAXED, __HIP_MEMORY_SCOPE_SYSTEM);
        tk = __builtin_amdgcn_readfirstlane((unsigned)v) & 127u;
        ok = (int)(__builtin_amdgcn_readfirstlane((unsigned)(v >> 32)) - call) >= 0;
        if (ok || __builtin_amdgcn_s_memrealtime() - t0 > DEADLINE_TICKS) break;
        __builtin_amdgcn_s_sleep(2);
      }
      TS(ts7);
      u32x4 mk;
      asm volatile("global_load_dwordx4 %0, %1, off sc0 sc1\n\ts_waitcnt vmcnt(0)"
                   : "=&v"(mk) : "v"(myws + WS_MSK + (size_t)(par * 1024 + slot) * 1024 + lane * 16) : "memory");
      const int cnt = __popc(mk[0]) + __popc(mk[1]) + __popc(mk[2]) + __popc(mk[3]);
      const int inc = wave_scan(cnt);
      const unsigned G = ((unsigned)__builtin_amdgcn_readlane(inc, 63) + 17u) / 18u;
      unsigned a = (unsigned)(inc - cnt) + lw * L_STG;
      unsigned ad[32], sel[32];
#pragma unroll
      for (int i = 0; i < 32; ++i) {
        const unsigned m4 = (mk[i >> 3] >> (4 * (i & 7))) & 15u;
        sel[i] = *(LAS unsigned*)(lds + L_TBL + ((m4 << 2 | (a & 3u)) << 2));
        ad[i] = a & ~3u;
        a += __popc(m4);
      }
#pragma unroll
      for (int i = 0; i < 32; i += 2) {
        const unsigned a0 = ad[i], a1 = ad[i + 1], s0 = sel[i], s1 = sel[i + 1];
        ad[i] = odd ? a1 : a0; ad[i + 1] = odd ? a0 : a1; sel[i] = odd ? s1 : s0; sel[i + 1] = odd ? s0 : s1;
      }
#pragma unroll
      for (int i = 0; i < 32; ++i) asm volatile("" : "+v"(ad[i]), "+v"(sel[i]));
      TS(ts8);
      unsigned ts = 0;
      while (ok) {
        const unsigned fr_ = __hip_atomic_load((LAS unsigned*)(lds + L_FREE + lw * 4), __ATOMIC_ACQUIRE, __HIP_MEMORY_SCOPE_WORKGROUP);
        ts = __builtin_amdgcn_readfirstlane(__hip_atomic_load((LAS unsigned*)(lds + L_SCALE + 4), __ATOMIC_ACQUIRE, __HIP_MEMORY_SCOPE_WORKGROUP));
        if (__builtin_amdgcn_readfirstlane(fr_) != 0u && ts != 0u) break;
        if (__builtin_amdgcn_s_memrealtime() - t0 > DEADLINE_TICKS) ok = false;
        __builtin_amdgcn_s_sleep(1);
      }
      const unsigned tag = (call & 2u) ? 0x00800000u : 0x00000080u;
      unsigned char* vs = myws + WS_VAL + (size_t)(par * 1024 + slot) * 8192;
      for (unsigned h = 0; ok && h < G; h += 256) {
        unsigned vo[4], am[4][4], ax[4];
#pragma unroll
        for (int j = 0; j < 4; ++j) {
          const unsigned g0 = h + j * 64, gi = g0 + lane, st = min(G - min(G, g0), 64u) * 4u;
          vo[j] = min(gi, G - 1u) * 16u;
          const unsigned b0 = (unsigned)(size_t)lds + lw * L_STG;
#pragma unroll
          for (int k = 0; k < 4; ++k) { am[j][k] = b0 + (gi < G ? g0 * 16u + k * st : 16384u) + lane * 4u; asm volatile("" : "+v"(am[j][k])); }
          ax[j] = b0 + (gi < G ? G * 16u + gi * 2u : 20480u + lane * 2u);
          asm volatile("" : "+v"(vo[j]), "+v"(ax[j]));
        }
        if (h == 0) {
          const int dt = (int)((tk * G * PACE) / 2580u) + GATE;
          while ((int)(((unsigned)__builtin_amdgcn_s_memrealtime() << 2) - (ts & ~3u)) < dt * 4) __builtin_amdgcn_s_sleep(1);
        }
        u32x4 d[4];
        for (;;) {
          asm volatile("global_load_dwordx4 %0, %4, %8 sc0 sc1\n\tglobal_load_dwordx4 %1, %5, %8 sc0 sc1\n\t"
                       "global_load_dwordx4 %2, %6, %8 sc0 sc1\n\tglobal_load_dwordx4 %3, %7, %8 sc0 sc1\n\ts_waitcnt vmcnt(0)"
                       : "=&v"(d[0]), "=&v"(d[1]), "=&v"(d[2]), "=&v"(d[3])
                       : "v"(vo[0]), "v"(vo[1]), "v"(vo[2]), "v"(vo[3]), "s"(vs) : "memory");
          const unsigned x = ((d[0][0] ^ tag) | (d[1][0] ^ tag) | (d[2][0] ^ tag) | (d[3][0] ^ tag)) & 0x00800080u;
          if (__all(x == 0u)) break;
          if (__builtin_amdgcn_s_memrealtime() - t0 > DEADLINE_TICKS) { ok = false; break; }
        }
#pragma unroll
        for (int j = 0; j < 4; ++j) {
          unsigned t1, t2, t3;
          asm volatile("v_lshrrev_b32 %0, 1, %4\n\tv_lshrrev_b32 %1, 2, %5\n\tv_bfi_b32 %0, %12, %3, %0\n\tv_lshrrev_b32 %2, 3, %6\n\t"
                       "v_bfi_b32 %0, %13, %0, %1\n\tv_and_b32 %1, %16, %3\n\tds_write_b32 %7, %1\n\tv_bfi_b32 %0, %14, %0, %2\n\t"
                       "v_and_b32 %2, %16, %4\n\tds_write_b32 %8, %2\n\tv_lshrrev_b32 %1, 12, %0\n\tv_bfi_b32 %0, %15, %0, %1\n\t"
                       "v_and_b32 %1, %16, %5\n\tds_write_b32 %9, %1\n\tv_and_b32 %2, %16, %6\n\tds_write_b32 %10, %2\n\t"
                       "v_and_b32 %0, %16, %0\n\tds_write_b8 %11, %0\n\tds_write_b8_d16_hi %11, %0 offset:1"
                       : "=&v"(t1), "=&v"(t2), "=&v"(t3)
                       : "v"(d[j][0]), "v"(d[j][1]), "v"(d[j][2]), "v"(d[j][3]), "v"(am[j][0]), "v"(am[j][1]), "v"(am[j][2]), "v"(am[j][3]), "v"(ax[j]),
                         "s"(0x80808080u), "s"(0xc0c0c0c0u), "s"(0xe0e0e0e0u), "s"(0xf0f0f0f0u), "s"(0x7f7f7f7fu) : "memory");
        }
      }
      if (ok) {
        u32x2 d[32];
#pragma unroll
        for (int i = 0; i < 32; ++i) { d[i][0] = *(LAS unsigned*)(lds + ad[i]); d[i][1] = *(LAS unsigned*)(lds + ad[i] + 4); }
#pragma unroll
        for (int mi = 0; mi < 8; ++mi) {
          u32x4 o;
#pragma unroll
          for (int k = 0; k < 4; ++k) o[k] = __builtin_amdgcn_perm(d[mi * 4 + k][1], d[mi * 4 + k][0], sel[mi * 4 + k]);
          STORE_OUT(mi, o);
        }
        TS(ts11);
        TSOUT(16 + lw, ts10);
        if (lw == 0) TSOUT(9, ts11 - ts10);
        if (lw == 0) TSOUT(11, npoll);
      } else {
        const u32x4 o = {0x7f7f7f7fu, 0x7f7f7f7fu, 0x7f7f7f7fu, 0x7f7f7f7fu};
#pragma unroll
        for (int mi = 0; mi < 8; ++mi) STORE_OUT(mi, o);
      }
    }
  } else {
    if (wid == 0) STAMP(0);
    const int wm = wid >> 1, wn = wid & 1;
    const int qq = (lane & 15) >> 2, pp = lane & 3;
    const unsigned xa0 = wm * 128 * 128 + fr * 128 + ((g ^ (fr >> 1)) * 16);
    unsigned wa[4];
#pragma unroll
    for (int ni = 0; ni < 4; ++ni)
      wa[ni] = W_BASE + (8 * g + qq) * 256 + (((wn * 8 + 2 * pp + (ni >> 1)) ^ (8 * (g & 1) + qq)) * 16) + (((ni ^ pp) & 1) * 8);
    f32x4 acc[8][4];
#pragma unroll
    for (int i = 0; i < 8; ++i)
#pragma unroll
      for (int j = 0; j < 4; ++j) acc[i][j] = (f32x4){0.f, 0.f, 0.f, 0.f};
    bf16x8 xA[8], wA[4], xB[8], wB[4];
    unsigned char* peerws = p.ws[peer];
    asm volatile("" : "+s"(peerws));
    BARRIER_L();
    if (wid == 0) STAMP(1);
#pragma unroll
    for (int ni = 0; ni < 4; ++ni) wA[ni] = ldw(lds, wa[ni], 0);
#pragma unroll
    for (int mi = 0; mi < 8; ++mi) xA[mi] = ldx(lds, xa0, mi * 2048);
    unsigned xoff = 0, woff = 0, cw[4], cx = xa0 ^ 64;
#pragma unroll
    for (int ni = 0; ni < 4; ++ni) cw[ni] = wa[ni];
    int tl = NT;
#define MF(W_, X_, mi, ni) acc[mi][ni] = __builtin_amdgcn_mfma_f32_16x16x32_bf16(W_[ni], X_[mi], acc[mi][ni], 0, 0, 0)
#define SB() __builtin_amdgcn_sched_barrier(0)
    do {
#pragma unroll
      for (int ni = 0; ni < 4; ++ni) wB[ni] = ldw(lds, cw[ni], 8192);
#pragma unroll
      for (int mi = 0; mi < 8; ++mi) xB[mi] = ldx(lds, cx, mi * 2048);
#pragma unroll
      for (int mi = 0; mi < 4; ++mi)
#pragma unroll
        for (int ni = 0; ni < 4; ++ni) MF(wA, xA, mi, ni);
#pragma unroll
      for (int i = 0; i < 16; ++i) { __builtin_amdgcn_sched_group_barrier(0x8, 1, 0); __builtin_amdgcn_sched_group_barrier(0x100, 1, 0); }
      SB(); MF(wA, xA, 4, 0); SB();
      xoff = (xoff == (XS - 1) * X_SLOT) ? 0 : xoff + X_SLOT; asm volatile("" : "+s"(xoff));
      SB(); MF(wA, xA, 4, 1); SB();
      woff = (woff + W_SLOT) & (WS * W_SLOT - 1); asm volatile("" : "+s"(woff));
      cx = xa0 + xoff; asm volatile("" : "+v"(cx));
      SB(); MF(wA, xA, 4, 2); SB();
      cw[0] = wa[0] + woff; cw[1] = wa[1] + woff; asm volatile("" : "+v"(cw[0]), "+v"(cw[1]));
      SB(); MF(wA, xA, 4, 3); SB();
      cw[2] = wa[2] + woff; cw[3] = wa[3] + woff; asm volatile("" : "+v"(cw[2]), "+v"(cw[3]));
      SB();
#pragma unroll
      for (int mi = 5; mi < 8; ++mi)
#pragma unroll
        for (int ni = 0; ni < 4; ++ni) MF(wA, xA, mi, ni);
      SB();
      BARRIER_L();
      SB();
#pragma unroll
      for (int ni = 0; ni < 4; ++ni) wA[ni] = ldw(lds, cw[ni], 0);
#pragma unroll
      for (int mi = 0; mi < 8; ++mi) xA[mi] = ldx(lds, cx, mi * 2048);
#pragma unroll
      for (int mi = 0; mi < 4; ++mi)
#pragma unroll
        for (int ni = 0; ni < 4; ++ni) MF(wB, xB, mi, ni);
#pragma unroll
      for (int i = 0; i < 16; ++i) { __builtin_amdgcn_sched_group_barrier(0x8, 1, 1); __builtin_amdgcn_sched_group_barrier(0x100, 1, 1); }
      SB(); MF(wB, xB, 4, 0); SB();
      cx ^= 64; tl -= 1; asm volatile("" : "+v"(cx), "+s"(tl));
      SB(); MF(wB, xB, 4, 1); MF(wB, xB, 4, 2); MF(wB, xB, 4, 3);
#pragma unroll
      for (int mi = 5; mi < 8; ++mi)
#pragma unroll
        for (int ni = 0; ni < 4; ++ni) MF(wB, xB, mi, ni);
      SB();
    } while (tl != 0);
    if (wid == 0) STAMP(2);
    const unsigned call = drawn(cdraw);
    const unsigned par = call & 1u;
    {
      int mx = 0;
#pragma unroll
      for (int mi = 0; mi < 8; ++mi)
#pragma unroll
        for (int ni = 0; ni < 4; ++ni) {
          mx = max(max(mx, __float_as_int(acc[mi][ni][0])), __float_as_int(acc[mi][ni][1]));
          mx = max(max(mx, __float_as_int(acc[mi][ni][2])), __float_as_int(acc[mi][ni][3]));
        }
      mx = wave_max63(mx);
      if (lane == 63)
        __hip_atomic_store((GAS u64*)(myws + WS_AGG) + par * (NWG * 4) + bid * 4 + wid, ((u64)call << 32) | (unsigned)mx, __ATOMIC_RELAXED, __HIP_MEMORY_SCOPE_AGENT);
    }
    __builtin_amdgcn_sched_barrier(0);
    const u64 t0 = __builtin_amdgcn_s_memrealtime();
    const unsigned slot = (rank * 32 + q) * 4 + wid;
    unsigned G = 0, Gs = 0, tk = 0, tg[8];
    f32x2 ex[8];
    if (!local) {
      unsigned gp;
      unsigned char* gw = myws + WS_CNT + bid * 128 + 32 + (par * 4 + wid) * 4;
      draw(tk, myws + WS_TKT + (xcd * 8 + peer) * 128);
      asm volatile("global_load_dword %0, %1, off sc1" : "=v"(gp) : "v"(gw) : "memory");
      acc[7][3][3] = __int_as_float(max(__float_as_int(acc[7][3][3]), 0));
      u32x4 mk;
#pragma unroll
      for (int w = 0; w < 4; ++w) {
        unsigned m = 0;
#pragma unroll
        for (int i = 31; i >= 0; --i) m = __builtin_amdgcn_alignbit(m, __float_as_uint(acc[2 * w + (i >> 4)][(i >> 2) & 3][i & 3]), 31);
        mk[w] = ~m;
      }
      TS(tsa);
      STORE_SYS(peerws + WS_MSK + (size_t)(par * 1024 + slot) * 1024 + lane * 16, mk);
      const int cnt = __popc(mk[0]) + __popc(mk[1]) + __popc(mk[2]) + __popc(mk[3]);
      const int inc = wave_scan(cnt);
      G = ((unsigned)__builtin_amdgcn_readlane(inc, 63) + 17u) / 18u;
      unsigned a = (unsigned)(size_t)lds + wid * L_STG + (inc - cnt) * 4;
#pragma unroll
      for (int i = 0; i < 32; ++i) {
        const f32x4 v = acc[i >> 2][i & 3];
        int sg;
        unsigned r1, r2, r3;
        asm volatile("v_ashrrev_i32 %1, 31, %5\n\tv_lshl_add_u32 %2, %1, 2, %0\n\tds_write_b32 %0, %5 offset:%9\n\t"
                     "v_ashrrev_i32 %1, 31, %6\n\tv_lshl_add_u32 %3, %1, 2, %2\n\tds_write_b32 %2, %6 offset:%9+4\n\t"
                     "v_ashrrev_i32 %1, 31, %7\n\tv_lshl_add_u32 %4, %1, 2, %3\n\tds_write_b32 %3, %7 offset:%9+8\n\t"
                     "v_ashrrev_i32 %1, 31, %8\n\tv_lshl_add_u32 %0, %1, 2, %4\n\tds_write_b32 %4, %8 offset:%9+12"
                     : "+v"(a), "=&v"(sg), "=&v"(r1), "=&v"(r2), "=&v"(r3) : "v"(v[0]), "v"(v[1]), "v"(v[2]), "v"(v[3]), "n"(16 * i) : "memory");
      }
      TS(tsc);
      tk = (15u - ((drawn(tk) - 1u) & 15u)) * 8 + xcd;
      asm volatile("" : "+v"(gp));
      TS(tsd);
      if (lane == 0) {
        __hip_atomic_store((GAS u64*)(peerws + WS_FLG + slot * 128), ((u64)call << 32) | tk, __ATOMIC_RELAXED, __HIP_MEMORY_SCOPE_SYSTEM);
        __hip_atomic_store((GAS unsigned*)gw, G, __ATOMIC_RELAXED, __HIP_MEMORY_SCOPE_AGENT);
      }
      Gs = max(G, min((unsigned)__builtin_amdgcn_readfirstlane(gp), 512u));
      const unsigned tag = (call & 2u) ? 0x00800000u : 0x00000080u;
#pragma unroll
      for (int j = 0; j < 8; ++j) {
        tg[j] = j * 64 + lane < G ? tag : 0u;
        if (j * 64 < G) {
#pragma unroll
          for (int k = 0; k < 4; ++k)
            acc[j][k] = *(LAS f32x4*)(lds + wid * L_STG + j * 4096 + (k * min(G - j * 64, 64u) + lane) * 16);
          ex[j] = *(LAS f32x2*)(lds + wid * L_STG + min(G * 64 + (j * 64 + lane) * 8, (unsigned)(L_STG - 8)));
        } else {
          ex[j] = (f32x2){0.f, 0.f};
        }
      }
      asm volatile("s_waitcnt lgkmcnt(0)" ::: "memory");
      if (lane == 0) __hip_atomic_store((LAS unsigned*)(lds + L_FREE + wid * 4), 1u, __ATOMIC_RELEASE, __HIP_MEMORY_SCOPE_WORKGROUP);
    }
    if (wid == 0) STAMP(5);
    float inv; unsigned ts;
    if (wid == 0) {
      GAS u64* gl = (GAS u64*)(myws + WS_GLB + par * 128);
      int mv; bool bad;
      for (;;) {
        const u64 v = __hip_atomic_load(gl + (lane & 7), __ATOMIC_RELAXED, __HIP_MEMORY_SCOPE_SYSTEM);
        mv = (int)(unsigned)v;
        bad = !__all((unsigned)(v >> 32) == call);
        if (!bad || __builtin_amdgcn_s_memrealtime() - t0 > DEADLINE_TICKS) break;
        __builtin_amdgcn_s_sleep(1);
      }
      ts = ((unsigned)__builtin_amdgcn_s_memrealtime() << 2) | 1u;
      mv = max(mv, DPP0(mv, 0x111, 0xf)); mv = max(mv, DPP0(mv, 0x112, 0xf)); mv = max(mv, DPP0(mv, 0x114, 0xf));
      const float amax = __int_as_float(__builtin_amdgcn_readlane(mv, 7));
      const float scale = amax / 448.0f;
      inv = amax > 0.f ? 1.0f / scale : 0.f;
      if (bad) inv = __int_as_float(0x7fc00000);
      if (lane == 0)
        __hip_atomic_store((LAS u64*)(lds + L_SCALE), ((u64)ts << 32) | __float_as_uint(inv), __ATOMIC_RELEASE, __HIP_MEMORY_SCOPE_WORKGROUP);
    } else {
      u64 v;
      for (;;) {
        v = __hip_atomic_load((LAS u64*)(lds + L_SCALE), __ATOMIC_ACQUIRE, __HIP_MEMORY_SCOPE_WORKGROUP);
        ts = __builtin_amdgcn_readfirstlane((unsigned)(v >> 32));
        if (ts != 0u) break;
        if (__builtin_amdgcn_s_memrealtime() - t0 > 2 * DEADLINE_TICKS) { v = 0x7fc00000u; tk = 0; break; }
        __builtin_amdgcn_s_sleep(1);
      }
      inv = __uint_as_float((unsigned)v);
    }
    if (wid == 0) STAMP(3);
    if (local) {
#pragma unroll
      for (int mi = 0; mi < 8; ++mi) {
        unsigned d[4];
#pragma unroll
        for (int ni = 0; ni < 4; ++ni) {
          f32x4 v;
#pragma unroll
          for (int r = 0; r < 4; ++r) v[r] = __int_as_float(max(__float_as_int(acc[mi][ni][r]), 0));
          d[ni] = pack4(v * inv);
        }
        u32x4 o;
        o[0] = odd ? d[1] : d[0]; o[1] = odd ? d[0] : d[1]; o[2] = odd ? d[3] : d[2]; o[3] = odd ? d[2] : d[3];
        STORE_OUT(mi, o);
      }
    } else {
      unsigned char* vd = peerws + WS_VAL + (size_t)(par * 1024 + slot) * 8192 + lane * 16;
      auto conv = [&](int j) {
        const f32x2 xe = ex[j] * inv;
        const unsigned e = (unsigned)__builtin_amdgcn_cvt_pk_fp8_f32(xe[0], xe[1], __float_as_int(xe[0]), false);
        const unsigned E = bfi(0x00800080u, tg[j], __builtin_amdgcn_perm(e << 4, e, 0x05010400u));
        u32x4 o;
#pragma unroll
        for (int k = 0; k < 4; ++k) o[k] = bfi(0x80808080u, E << k, pack4(acc[j][k] * inv));
        return o;
      };
      if (tk < 16) {
#pragma unroll
        for (int j = 0; j < 8; ++j)
          if (j * 64 < Gs) { const u32x4 o = conv(j); if (j * 64 + lane < Gs) STORE_SYS(vd + j * 1024, o); }
      } else {
        u32x4 o[8];
#pragma unroll
        for (int j = 0; j < 8; ++j) { if (j * 64 < Gs) o[j] = conv(j); else o[j] = (u32x4){0u, 0u, 0u, 0u}; asm volatile("" : "+v"(o[j])); }
        const int dt = (int)((tk * G * PACE) / 2580u);
        while ((int)(((unsigned)__builtin_amdgcn_s_memrealtime() << 2) - (ts & ~3u)) < dt * 4) __builtin_amdgcn_s_sleep(1);
#pragma unroll
        for (int j = 0; j < 8; ++j)
          if (j * 64 < Gs) { if (j * 64 + lane < Gs) STORE_SYS(vd + j * 1024, o[j]); }
      }
    }
  }
}

}

extern "C" void kernel_launch(int rank, int nranks, ncclComm_t comm, hipStream_t stream,
                              void* const* d_in, const int* in_sizes, int n_in,
                              void* d_out, int out_size, const int* in_dtypes, int out_dtype,
                              void* const* d_out_all, void* const* d_ws_all, size_t ws_bytes) {
  (void)comm; (void)in_dtypes; (void)out_dtype; (void)d_out_all;
  if (nranks != NR || n_in != 2 || in_sizes[0] != MLOC * KDIM || in_sizes[1] != KDIM * NDIM ||
      out_size != NR * MLOC * NSH || ws_bytes < WS_END || (unsigned)rank >= (unsigned)NR) {
    fprintf(stderr, "unexpected problem shape\n");
    return;
  }
  Params p;
  p.x = (const unsigned short*)d_in[0];
  p.w = (const unsigned short*)d_in[1];
  p.out = (unsigned char*)d_out;
  for (int i = 0; i < NR; ++i) p.ws[i] = (unsigned char*)d_ws_all[i];
  p.rank = rank; p.pad = 0;
  hipLaunchKernelGGL(fused_gemm_a2a, dim3(NWG), dim3(512), 0, stream, p);
  HIP_CHECK(hipGetLastError());
}
```

```cpp
#include <hip/hip_runtime.h>
#include <rccl/rccl.h>
#include <cstddef>
#include <cstdio>

#define HIP_CHECK(expr) do { hipError_t _e = (expr); if (_e != hipSuccess) { \
    fprintf(stderr, "HIP error %s at %s:%d\n", hipGetErrorString(_e), __FILE__, __LINE__); \
    return; } } while (0)

namespace {

typedef short bf16x8 __attribute__((ext_vector_type(8)));
typedef short s16x4 __attribute__((ext_vector_type(4)));
typedef float f32x4 __attribute__((ext_vector_type(4)));
typedef float f32x2 __attribute__((ext_vector_type(2)));
typedef unsigned u32x4 __attribute__((ext_vector_type(4)));
typedef unsigned u32x2 __attribute__((ext_vector_type(2)));
typedef unsigned u32x8 __attribute__((ext_vector_type(8)));
typedef unsigned long long u64;
#define LAS __attribute__((address_space(3)))
#define GAS __attribute__((address_space(1)))

constexpr int NR = 8;
constexpr int MLOC = 512, KDIM = 4096, NDIM = 16384, NSH = NDIM / NR;
constexpr int BM = 256, BN = 128, BK = 64, NT = KDIM / BK;
constexpr int NWG = (MLOC / BM) * (NDIM / BN);
constexpr int X_SLOT = BM * BK * 2, W_SLOT = BK * BN * 2, XS = 3, WS = 4;
constexpr int W_BASE = XS * X_SLOT, LDS_BYTES = W_BASE + WS * W_SLOT;
constexpr int L_STG = 32768, L_TBL = 4 * L_STG, L_SCALE = L_TBL + 256, L_FREE = L_SCALE + 16;

constexpr size_t WS_CNT = 0;
constexpr size_t WS_AGG = WS_CNT + NWG * 128;
constexpr size_t WS_GLB = WS_AGG + 2 * NWG * 4 * 8;
constexpr size_t WS_TKT = WS_GLB + 2 * 128;
constexpr size_t WS_FLG = WS_TKT + 64 * 128;
constexpr size_t WS_MSK = 1u << 20;
constexpr size_t WS_VAL = WS_MSK + (2u << 20);
constexpr size_t WS_END = WS_VAL + (16u << 20);
static_assert(WS_FLG + NR * 32 * 4 * 128 <= WS_MSK, "layout");

constexpr int PACE = 70, GATE = 40;
constexpr int NAGG = 4;
constexpr int PFD = 6;
constexpr int W_NT = 2;
constexpr u64 DEADLINE_TICKS = 50000000ull;

#define X_(m, k) (32 + 2 * (m) + (k))
#define W_R(j) (64 + (j))
constexpr unsigned char ACT[2][64] = {
  {X_(2,0),0,X_(2,1),W_R(0),X_(3,0),0,X_(3,1),W_R(1),  X_(4,0),0,X_(4,1),W_R(2),W_R(3),0,W_R(4),0,  X_(5,0),0,X_(5,1),W_R(5),W_R(6),0,W_R(7),0,
   X_(6,0),0,X_(6,1),W_R(8),W_R(9),0,W_R(10),0,  X_(7,0),0,X_(7,1),W_R(11),W_R(12),1,W_R(13),2,  W_R(14),0,W_R(15),3,4,0,0,5,
   X_(8,0),0,X_(8,1),0,X_(9,0),0,X_(9,1),0,  0,0,6,0,0,0,0,0},
  {X_(2,0),0,X_(2,1),0,X_(3,0),0,X_(3,1),0,  X_(4,0),0,X_(4,1),0,0,0,0,0,  X_(5,0),0,X_(5,1),0,0,0,0,5,
   X_(6,0),W_R(0),X_(6,1),W_R(1),W_R(2),W_R(3),W_R(4),W_R(5),  X_(7,0),W_R(6),X_(7,1),W_R(7),W_R(8),W_R(9),W_R(10),W_R(11),  W_R(12),W_R(13),W_R(14),W_R(15),1,2,3,4,
   0,0,0,5,X_(8,0),0,X_(8,1),0,  X_(9,0),0,X_(9,1),0,0,0,0,0}};
struct Params {
  const unsigned short* x;
  const unsigned short* w;
  unsigned char* out;
  unsigned char* my;
  int rank;
  int pad;
  unsigned char* ws[NR];
};
static_assert(offsetof(Params, my) == 0x18 && offsetof(Params, rank) == 0x20, "read by offset in the kernel");

#define BARRIER_L() asm volatile("s_waitcnt lgkmcnt(0)\n\ts_barrier" ::: "memory")
#define BARRIER_V(n) asm volatile("s_waitcnt vmcnt(" #n ")\n\ts_barrier" ::: "memory")
#define STORE_SYS(ptr, val) asm volatile("global_store_dwordx4 %0, %1, off sc0 sc1\n\ts_nop 1" :: "v"(ptr), "v"(val) : "memory")
#define OBITS " sc0 sc1"
#define STORE_OUT(mi, val) do { if ((mi) == 0) asm volatile("s_nop 4\n\tbuffer_store_dwordx4 %0, %1, %2, %3 offen" OBITS "\n\ts_nop 1" :: "v"(val), "v"(ovo), "s"(rout), "s"((mi) * 16 * NSH) : "memory"); \
  else asm volatile("buffer_store_dwordx4 %0, %1, %2, %3 offen" OBITS "\n\ts_nop 1" :: "v"(val), "v"(ovo), "s"(rout), "s"((mi) * 16 * NSH) : "memory"); } while (0)

__device__ __forceinline__ bf16x8 ldx(LAS char* lds, unsigned addr, int imm) { return *(LAS bf16x8*)(lds + addr + imm); }
#define DPP0(v, ctrl, rm) __builtin_amdgcn_update_dpp(0, (v), (ctrl), (rm), 0xf, false)
__device__ __forceinline__ int wave_max63(int v) {
  v = max(v, DPP0(v, 0x111, 0xf)); v = max(v, DPP0(v, 0x112, 0xf)); v = max(v, DPP0(v, 0x114, 0xf)); v = max(v, DPP0(v, 0x118, 0xf));
  v = max(v, DPP0(v, 0x142, 0xa)); v = max(v, DPP0(v, 0x143, 0xc));
  return v;
}
__device__ __forceinline__ int wave_scan(int v) {
  v += DPP0(v, 0x111, 0xf); v += DPP0(v, 0x112, 0xf); v += DPP0(v, 0x114, 0xf); v += DPP0(v, 0x118, 0xf);
  v += DPP0(v, 0x142, 0xa); v += DPP0(v, 0x143, 0xc);
  return v;
}
__device__ __forceinline__ void draw(unsigned& c, unsigned char* cnt) {
  asm volatile("s_mov_b64 exec, 1\n\tglobal_atomic_add %0, %1, %2, off sc0\n\ts_mov_b64 exec, -1" : "=v"(c) : "v"(cnt), "v"(1u) : "memory");
}
__device__ __forceinline__ unsigned drawn(unsigned& c) {
  unsigned r;
  asm volatile("s_waitcnt vmcnt(0)\n\tv_readfirstlane_b32 %0, %1" : "=s"(r) : "v"(c) : "memory");
  return r + 1u;
}
__device__ __forceinline__ unsigned pack4(f32x4 v) {
  int r = __builtin_amdgcn_cvt_pk_fp8_f32(v[0], v[1], __float_as_int(v[0]), false);
  return (unsigned)__builtin_amdgcn_cvt_pk_fp8_f32(v[2], v[3], r, true);
}
__device__ __forceinline__ unsigned bfi(unsigned m, unsigned a, unsigned b) { return (m & a) | (~m & b); }

__device__ __forceinline__ int panel_of(int pj, int xcd) {
  const int pn = (pj >> 1) * 16 + (xcd >> 1) * 4 + (pj & 1) * 2 + (~xcd & 1);
  return (pn & 30) == 30 ? pn ^ 1 : pn;
}
__global__ __launch_bounds__(512, 2) __attribute__((amdgpu_num_sgpr(96))) void fused_gemm_a2a(const Params p) {
  __shared__ __attribute__((aligned(16))) char smem_[LDS_BYTES];
  LAS char* lds = (LAS char*)smem_;
  u32x8 ka; u32x2 kb;
  asm volatile("s_load_dwordx8 %0, %2, 0x0\n\ts_load_dwordx2 %1, %2, 0x20" : "=&s"(ka), "=&s"(kb) : "s"(__builtin_amdgcn_kernarg_segment_ptr()));
  __builtin_amdgcn_sched_barrier(0);
  const int bid = blockIdx.x;
  const int xcd = bid & 7, idx = bid >> 3;
  const int pj = idx >> 1;
  const int panel = panel_of(pj, xcd), mt = idx & 1;
  const unsigned peer = (unsigned)(panel >> 4) & (NR - 1);
  const int q = (panel & 15) * 2 + mt;
  const int wid = __builtin_amdgcn_readfirstlane(threadIdx.x >> 6);
  const int lane = threadIdx.x & 63;
  const int fr = lane & 15, g = lane >> 4;
  const bool odd = g & 1;
  const int wv4 = wid & 3;
  unsigned ovo = (unsigned)((peer * MLOC + mt * BM + (wv4 >> 1) * 128 + fr) * NSH + (panel & 15) * BN + (wv4 & 1) * 64 + g * 16);
  unsigned xv = (unsigned)((lane >> 3) * (KDIM * 2) + (((lane & 7) ^ (4 * (wid & 1) + (lane >> 4))) * 16));
  unsigned wv = (unsigned)((lane >> 4) * (NDIM * 2) + (((lane & 15) ^ ((lane >> 4) + 8 * ((wid >> 1) & 1))) * 16) + panel * (BN * 2));
  asm volatile("" : "+v"(xv), "+v"(wv), "+v"(ovo));
  __builtin_amdgcn_sched_barrier(0);
  asm volatile("s_waitcnt lgkmcnt(0)" : "+s"(ka), "+s"(kb));
  const unsigned short* gx = (const unsigned short*)(GAS const unsigned short*)(((u64)ka[1] << 32) | ka[0]);
  const unsigned short* gw = (const unsigned short*)(GAS const unsigned short*)(((u64)ka[3] << 32) | ka[2]);
  unsigned char* gout = (unsigned char*)(GAS unsigned char*)(((u64)ka[5] << 32) | ka[4]);
  unsigned char* myws = (unsigned char*)(GAS unsigned char*)(((u64)ka[7] << 32) | ka[6]);
  const int rank = (int)kb[0];
  const bool local = peer == (unsigned)rank;
  const auto rout = __builtin_amdgcn_make_buffer_rsrc((void*)gout, 0, NR * MLOC * NSH, 0x00020000);
  unsigned cdraw;
  draw(cdraw, myws + WS_CNT + bid * 128 + wid * 4);

  if (__builtin_expect(wid >= 4, 1)) {
    const int lw = wid - 4;
    auto rx = __builtin_amdgcn_make_buffer_rsrc((void*)(gx + (size_t)mt * BM * KDIM), 0, BM * KDIM * 2, 0x00020000);
    auto rw = __builtin_amdgcn_make_buffer_rsrc((void*)gw, 0, KDIM * NDIM * 2, 0x00020000);
    const unsigned xj0 = (lw >> 1) * 16 + (lw & 1);
    const unsigned wj0 = lw;
    auto issue_x = [&](int t, int slot) {
      const unsigned so = (t < NT) ? (unsigned)(t * BK * 2) : 0x80000000u;
#pragma unroll
      for (int jj = 0; jj < 8; ++jj) {
        const unsigned j = xj0 + 2 * jj;
        __builtin_amdgcn_raw_ptr_buffer_load_lds(rx, (LAS void*)(lds + slot * X_SLOT + j * 1024), 16, xv, so + j * 8 * (KDIM * 2), 0, 0);
      }
    };
    auto issue_w = [&](int t, int slot) {
      const unsigned so = (t < NT) ? (unsigned)(t * BK * (NDIM * 2)) : 0x80000000u;
#pragma unroll
      for (int jj = 0; jj < 4; ++jj) {
        const unsigned j = wj0 + 4 * jj;
        __builtin_amdgcn_raw_ptr_buffer_load_lds(rw, (LAS void*)(lds + W_BASE + slot * W_SLOT + j * 1024), 16, wv, so + j * 4 * (NDIM * 2), 0, W_NT);
      }
    };
    issue_w(0, 0); issue_x(0, 0);
    issue_w(1, 1);
    issue_w(2, 2);
    issue_x(1, 1);
    unsigned warm = 0;
    {
      auto rxa = __builtin_amdgcn_make_buffer_rsrc((void*)gx, 0, MLOC * KDIM * 2, 0x00020000);
      const unsigned a = (unsigned)(bid * 16384 + lw * 4096 + lane * 64);
      asm volatile("buffer_load_dword %0, %1, %2, 0 offen" : "+v"(warm) : "v"(a), "s"(rxa) : "memory");
    }
    const char* pq = (const char*)gw + (size_t)((idx & 7) * 4 + lw) * (NDIM * 2) + 60;
    unsigned ptab;
    {
      const int u = lane >> 1, gq = pj >> 2;
      const int T = min(2 * u + PFD, NT - 1);
      ptab = (unsigned)(T * BK + (lane & 1) * 32) * (NDIM * 2) + panel_of(gq * 4 + (xcd == 7 ? 3 : (u & 1) * 2), xcd) * (BN * 2);
      asm volatile("" : "+v"(ptab));
    }
    const bool pf = xcd == 7 ? (pj >> 2) != 1 : (xcd & 1) == 0;
    auto pref = [&](int t) {
      if (pf) {
        const unsigned o = __builtin_amdgcn_readlane(ptab, t);
        unsigned c;
        asm volatile("s_getreg_b32 %0, hwreg(HW_REG_IB_STS, 8, 4)\n\ts_cmp_gt_u32 %0, 11\n\ts_cbranch_scc1 1f\n\t"
                     "s_load_dwordx2 s[100:101], %1, %2 offset:0x0\n\ts_load_dwordx2 s[100:101], %1, %2 offset:0x80\n1:" : "=&s"(c) : "s"(pq), "s"(o) : "memory", "scc", "s100", "s101");
      }
    };
    unsigned char* lanews;
    {
      unsigned char* qq[NR];
#pragma unroll
      for (int i = 0; i < NR; ++i) qq[i] = p.ws[i];
      asm volatile("" : "+s"(qq[0]), "+s"(qq[1]), "+s"(qq[2]), "+s"(qq[3]), "+s"(qq[4]), "+s"(qq[5]), "+s"(qq[6]), "+s"(qq[7]));
      lanews = qq[0];
#pragma unroll
      for (int i = 1; i < NR; ++i) lanews = (lane & 7) == i ? qq[i] : lanews;
      asm volatile("" : "+v"(lanews));
    }
    unsigned tbl = 0;
    {
      unsigned pos = lane & 3;
#pragma unroll
      for (int k = 0; k < 4; ++k) {
        const unsigned keep = (lane >> (2 + k)) & 1;
        tbl |= (keep ? pos : 0x0cu) << (8 * k);
        pos += keep;
      }
      asm volatile("" : "+v"(tbl));
    }
    BARRIER_V(17);
    issue_w(3, 3); issue_x(2, 2);
    BARRIER_V(25);
    pref(0); issue_w(4, 0);
    int xs = 0, ws = 1;
    for (int t = 1; t < NT; ++t) {
      BARRIER_V(16);
      issue_x(t + 2, xs);
      issue_w(t + 4, ws);
      pref(t);
      xs = (xs == XS - 1) ? 0 : xs + 1; ws = (ws + 1) & (WS - 1);
    }
    asm volatile("s_waitcnt lgkmcnt(0)" ::: "memory");
    const unsigned call = drawn(cdraw);
    asm volatile("" : "+v"(warm));
    if (lw == 0) *(LAS unsigned*)(lds + L_TBL + lane * 4) = tbl;
    if (lw == 1 && lane < 8) *(LAS unsigned*)(lds + L_SCALE + lane * 4) = 0u;
    BARRIER_L();
    const unsigned par = call & 1u;
    const u64 t0 = __builtin_amdgcn_s_memrealtime();
    if (local) {
      if (q == 0 && lw < NAGG) {
        unsigned char* a = myws + WS_AGG + par * (NWG * 4 * 8) + lane * 16;
        if (lw == 1) __builtin_amdgcn_s_sleep(8);
        if (lw == 2) __builtin_amdgcn_s_sleep(16);
        if (lw == 3) __builtin_amdgcn_s_sleep(24);
        bool ok; int mv;
        for (;;) {
          u32x4 v[8];
          asm volatile("global_load_dwordx4 %0, %8, off sc1\n\tglobal_load_dwordx4 %1, %8, off offset:1024 sc1\n\t"
                       "global_load_dwordx4 %2, %8, off offset:2048 sc1\n\tglobal_load_dwordx4 %3, %8, off offset:3072 sc1\n\t"
                       "global_load_dwordx4 %4, %9, off sc1\n\tglobal_load_dwordx4 %5, %9, off offset:1024 sc1\n\t"
                       "global_load_dwordx4 %6, %9, off offset:2048 sc1\n\tglobal_load_dwordx4 %7, %9, off offset:3072 sc1\n\ts_waitcnt vmcnt(0)"
                       : "=&v"(v[0]), "=&v"(v[1]), "=&v"(v[2]), "=&v"(v[3]), "=&v"(v[4]), "=&v"(v[5]), "=&v"(v[6]), "=&v"(v[7])
                       : "v"(a), "v"(a + 4096) : "memory");
          unsigned d = 0; mv = 0;
#pragma unroll
          for (int i = 0; i < 8; ++i) { d |= (v[i][1] ^ call) | (v[i][3] ^ call); mv = max(mv, max((int)v[i][0], (int)v[i][2])); }
          ok = __all(d == 0);
          if (ok || __builtin_amdgcn_s_memrealtime() - t0 > DEADLINE_TICKS) break;
        }
        const unsigned m = (unsigned)__builtin_amdgcn_readlane(wave_max63(mv), 63);
        if (ok && lane < NR)
          __hip_atomic_store((GAS u64*)(lanews + WS_GLB + par * 128) + rank, ((u64)call << 32) | m, __ATOMIC_RELAXED, __HIP_MEMORY_SCOPE_SYSTEM);
      }
    } else {
      const unsigned slot = (peer * 32 + q) * 4 + lw;
      GAS u64* flag = (GAS u64*)(myws + WS_FLG + slot * 128);
      bool ok; unsigned tk;
      for (;;) {
        const u64 v = __hip_atomic_load(flag, __ATOMIC_RELAXED, __HIP_MEMORY_SCOPE_SYSTEM);
        tk = __builtin_amdgcn_readfirstlane((unsigned)v) & 127u;
        ok = (int)(__builtin_amdgcn_readfirstlane((unsigned)(v >> 32)) - call) >= 0;
        if (ok || __builtin_amdgcn_s_memrealtime() - t0 > DEADLINE_TICKS) break;
        __builtin_amdgcn_s_sleep(2);
      }
      u32x4 mk;
      asm volatile("global_load_dwordx4 %0, %1, off sc0 sc1\n\ts_waitcnt vmcnt(0)"
                   : "=&v"(mk) : "v"(myws + WS_MSK + (size_t)(par * 1024 + slot) * 1024 + lane * 16) : "memory");
      const int cnt = __popc(mk[0]) + __popc(mk[1]) + __popc(mk[2]) + __popc(mk[3]);
      const int inc = wave_scan(cnt);
      const unsigned G = ((unsigned)__builtin_amdgcn_readlane(inc, 63) + 17u) / 18u;
      unsigned a = (unsigned)(inc - cnt) + lw * L_STG;
      unsigned ad[32], sel[32];
#pragma unroll
      for (int i = 0; i < 32; ++i) {
        const unsigned m4 = (mk[i >> 3] >> (4 * (i & 7))) & 15u;
        sel[i] = *(LAS unsigned*)(lds + L_TBL + ((m4 << 2 | (a & 3u)) << 2));
        ad[i] = a & ~3u;
        a += __popc(m4);
      }
#pragma unroll
      for (int i = 0; i < 32; i += 2) {
        const unsigned a0 = ad[i], a1 = ad[i + 1], s0 = sel[i], s1 = sel[i + 1];
        ad[i] = odd ? a1 : a0; ad[i + 1] = odd ? a0 : a1; sel[i] = odd ? s1 : s0; sel[i + 1] = odd ? s0 : s1;
      }
#pragma unroll
      for (int i = 0; i < 32; ++i) asm volatile("" : "+v"(ad[i]), "+v"(sel[i]));
      unsigned ts = 0;
      while (ok) {
        const unsigned fr_ = __hip_atomic_load((LAS unsigned*)(lds + L_FREE + lw * 4), __ATOMIC_ACQUIRE, __HIP_MEMORY_SCOPE_WORKGROUP);
        ts = __builtin_amdgcn_readfirstlane(__hip_atomic_load((LAS unsigned*)(lds + L_SCALE + 4), __ATOMIC_ACQUIRE, __HIP_MEMORY_SCOPE_WORKGROUP));
        if (__builtin_amdgcn_readfirstlane(fr_) != 0u && ts != 0u) break;
        if (__builtin_amdgcn_s_memrealtime() - t0 > DEADLINE_TICKS) ok = false;
        __builtin_amdgcn_s_sleep(1);
      }
      const unsigned tag = (call & 2u) ? 0x00800000u : 0x00000080u;
      unsigned char* vs = myws + WS_VAL + (size_t)(par * 1024 + slot) * 8192;
      for (unsigned h = 0; ok && h < G; h += 256) {
        unsigned vo[4], am[4][4], ax[4];
#pragma unroll
        for (int j = 0; j < 4; ++j) {
          const unsigned g0 = h + j * 64, gi = g0 + lane, st = min(G - min(G, g0), 64u) * 4u;
          vo[j] = min(gi, G - 1u) * 16u;
          const unsigned b0 = (unsigned)(size_t)lds + lw * L_STG;
#pragma unroll
          for (int k = 0; k < 4; ++k) { am[j][k] = b0 + (gi < G ? g0 * 16u + k * st : 16384u) + lane * 4u; asm volatile("" : "+v"(am[j][k])); }
          ax[j] = b0 + (gi < G ? G * 16u + gi * 2u : 20480u + lane * 2u);
          asm volatile("" : "+v"(vo[j]), "+v"(ax[j]));
        }
        if (h == 0) {
          const int dt = (int)((tk * G * PACE) / 2580u) + GATE;
          while ((int)(((unsigned)__builtin_amdgcn_s_memrealtime() << 2) - (ts & ~3u)) < dt * 4) __builtin_amdgcn_s_sleep(1);
        }
        u32x4 d[4];
        for (;;) {
          asm volatile("global_load_dwordx4 %0, %4, %8 sc0 sc1\n\tglobal_load_dwordx4 %1, %5, %8 sc0 sc1\n\t"
                       "global_load_dwordx4 %2, %6, %8 sc0 sc1\n\tglobal_load_dwordx4 %3, %7, %8 sc0 sc1\n\ts_waitcnt vmcnt(0)"
                       : "=&v"(d[0]), "=&v"(d[1]), "=&v"(d[2]), "=&v"(d[3])
                       : "v"(vo[0]), "v"(vo[1]), "v"(vo[2]), "v"(vo[3]), "s"(vs) : "memory");
          const unsigned x = ((d[0][0] ^ tag) | (d[1][0] ^ tag) | (d[2][0] ^ tag) | (d[3][0] ^ tag)) & 0x00800080u;
          if (__all(x == 0u)) break;
          if (__builtin_amdgcn_s_memrealtime() - t0 > DEADLINE_TICKS) { ok = false; break; }
        }
#pragma unroll
        for (int j = 0; j < 4; ++j) {
          unsigned t1, t2, t3;
          asm volatile("v_lshrrev_b32 %0, 1, %4\n\tv_lshrrev_b32 %1, 2, %5\n\tv_bfi_b32 %0, %12, %3, %0\n\tv_lshrrev_b32 %2, 3, %6\n\t"
                       "v_bfi_b32 %0, %13, %0, %1\n\tv_and_b32 %1, %16, %3\n\tds_write_b32 %7, %1\n\tv_bfi_b32 %0, %14, %0, %2\n\t"
                       "v_and_b32 %2, %16, %4\n\tds_write_b32 %8, %2\n\tv_lshrrev_b32 %1, 12, %0\n\tv_bfi_b32 %0, %15, %0, %1\n\t"
                       "v_and_b32 %1, %16, %5\n\tds_write_b32 %9, %1\n\tv_and_b32 %2, %16, %6\n\tds_write_b32 %10, %2\n\t"
                       "v_and_b32 %0, %16, %0\n\tds_write_b8 %11, %0\n\tds_write_b8_d16_hi %11, %0 offset:1"
                       : "=&v"(t1), "=&v"(t2), "=&v"(t3)
                       : "v"(d[j][0]), "v"(d[j][1]), "v"(d[j][2]), "v"(d[j][3]), "v"(am[j][0]), "v"(am[j][1]), "v"(am[j][2]), "v"(am[j][3]), "v"(ax[j]),
                         "s"(0x80808080u), "s"(0xc0c0c0c0u), "s"(0xe0e0e0e0u), "s"(0xf0f0f0f0u), "s"(0x7f7f7f7fu) : "memory");
        }
      }
      if (ok) {
        u32x2 d[32];
#pragma unroll
        for (int i = 0; i < 32; ++i) { d[i][0] = *(LAS unsigned*)(lds + ad[i]); d[i][1] = *(LAS unsigned*)(lds + ad[i] + 4); }
#pragma unroll
        for (int mi = 0; mi < 8; ++mi) {
          u32x4 o;
#pragma unroll
          for (int k = 0; k < 4; ++k) o[k] = __builtin_amdgcn_perm(d[mi * 4 + k][1], d[mi * 4 + k][0], sel[mi * 4 + k]);
          STORE_OUT(mi, o);
        }
      } else {
        const u32x4 o = {0x7f7f7f7fu, 0x7f7f7f7fu, 0x7f7f7f7fu, 0x7f7f7f7fu};
#pragma unroll
        for (int mi = 0; mi < 8; ++mi) STORE_OUT(mi, o);
      }
    }
  } else {
    const int wm = wid >> 1, wn = wid & 1;
    const int qq = (lane & 15) >> 2, pp = lane & 3;
    const unsigned xa0 = wm * 128 * 128 + fr * 128 + ((g ^ (fr >> 1)) * 16);
    unsigned wa[4];
#pragma unroll
    for (int ni = 0; ni < 4; ++ni)
      wa[ni] = W_BASE + (8 * g + qq) * 256 + (((wn * 8 + 2 * pp + (ni >> 1)) ^ (8 * (g & 1) + qq)) * 16) + (((ni ^ pp) & 1) * 8);
    f32x4 acc[8][4];
    bf16x8 xb[4][2];
    s16x4 wA[2][4][2], wB[2][4][2];
    unsigned char* peerws = p.ws[peer];
    asm volatile("" : "+s"(peerws));
    unsigned xoff = 0, woff = W_SLOT, cw[4], cx0 = xa0, cx1 = xa0 ^ 64;
#define WRD(W_, j) W_[((j) >> 1) & 1][(j) >> 2][(j) & 1] = __builtin_amdgcn_ds_read_tr16_b64_v4i16((LAS s16x4*)(lds + cw[(j) >> 2] + (((j) >> 1) & 1) * 8192 + ((j) & 1) * 1024))
#define XRD(m, k) xb[(m) & 3][k] = ldx(lds, (k) ? cx1 : cx0, ((m) & 7) * 2048)
#define SB() __builtin_amdgcn_sched_barrier(0)
#pragma unroll
    for (int ni = 0; ni < 4; ++ni) cw[ni] = wa[ni];
    BARRIER_L();
    XRD(0, 0); XRD(0, 1);
#pragma unroll
    for (int j = 0; j < 16; ++j) WRD(wA, j);
#pragma unroll
    for (int ni = 0; ni < 4; ++ni) cw[ni] = wa[ni] + W_SLOT;
    XRD(1, 0); XRD(1, 1);
    int tl = NT / 2 - 2;
    int mx = 0;
    auto step = [&](s16x4 (&W_)[2][4][2], s16x4 (&WN)[2][4][2], bool last, int tab, bool fin = false) {
#pragma unroll
      for (int sl = 0; sl < 64; ++sl) {
        const int i = sl >> 1, m = i >> 2, n = (m & 1) ? 3 - (i & 3) : (i & 3), k = (i ^ sl) & 1;
        if (tab == 1 && (sl & 1) == 0)
          asm volatile("v_mfma_f32_16x16x32_bf16 %0, %1, %2, 0" : "=&v"(acc[m][n])
                       : "v"(__builtin_shufflevector(W_[k][n][0], W_[k][n][1], 0, 1, 2, 3, 4, 5, 6, 7)), "v"(xb[m & 3][k]));
        else
        asm volatile("v_mfma_f32_16x16x32_bf16 %0, %1, %2, %0" : "+v"(acc[m][n])
                     : "v"(__builtin_shufflevector(W_[k][n][0], W_[k][n][1], 0, 1, 2, 3, 4, 5, 6, 7)), "v"(xb[m & 3][k]));
        SB();
        const int act = ACT[tab][sl];
        if (act >= 64) WRD(WN, act - 64);
        else if (act >= 32) XRD((act - 32) >> 1, act & 1);
        else if (act == 1) { xoff = (xoff == (XS - 1) * X_SLOT) ? 0 : xoff + X_SLOT; woff = (woff + W_SLOT) & (WS * W_SLOT - 1); asm volatile("" : "+s"(xoff), "+s"(woff)); }
        else if (act == 2) { cx0 = xa0 + xoff; cx1 = cx0 ^ 64; asm volatile("" : "+v"(cx0), "+v"(cx1)); }
        else if (act == 3) { cw[0] = wa[0] + woff; cw[1] = wa[1] + woff; asm volatile("" : "+v"(cw[0]), "+v"(cw[1])); }
        else if (act == 4) { cw[2] = wa[2] + woff; cw[3] = wa[3] + woff; asm volatile("" : "+v"(cw[2]), "+v"(cw[3])); }
        else if (act == 5) BARRIER_L();
        else if (act == 6 && last) { tl -= 1; asm volatile("" : "+s"(tl)); }
        if (fin && sl >= 10) {
          const int r = (sl - 10) >> 3, c = (sl - 10) & 7;
          asm volatile("v_max3_i32 %0, %0, %1, %2" : "+v"(mx) : "v"(acc[r][c >> 1][(c & 1) * 2]), "v"(acc[r][c >> 1][(c & 1) * 2 + 1]));
        }
        SB();
      }
    };
    step(wA, wB, false, 1); step(wB, wA, false, 0);
    do {
      step(wA, wB, false, 0);
      step(wB, wA, true, 0);
    } while (tl != 0);
    step(wA, wB, false, 0);
    step(wB, wA, false, 0, true);
    asm volatile("s_nop 7\n\ts_nop 7");
    SB();
    const unsigned call = drawn(cdraw);
    const unsigned par = call & 1u;
    {
#pragma unroll
      for (int i = 54; i < 64; ++i)
        mx = max(max(mx, __float_as_int(acc[i >> 3][(i & 7) >> 1][(i & 1) * 2])), __float_as_int(acc[i >> 3][(i & 7) >> 1][(i & 1) * 2 + 1]));
      mx = wave_max63(mx);
      if (lane == 63)
        __hip_atomic_store((GAS u64*)(myws + WS_AGG) + par * (NWG * 4) + bid * 4 + wid, ((u64)call << 32) | (unsigned)mx, __ATOMIC_RELAXED, __HIP_MEMORY_SCOPE_AGENT);
    }
    __builtin_amdgcn_sched_barrier(0);
    const u64 t0 = __builtin_amdgcn_s_memrealtime();
    const unsigned slot = (rank * 32 + q) * 4 + wid;
    unsigned G = 0, Gs = 0, tk = 0, tg[8];
    f32x2 ex[8];
    if (!local) {
      unsigned gp;
      unsigned char* gw = myws + WS_CNT + bid * 128 + 32 + (par * 4 + wid) * 4;
      draw(tk, myws + WS_TKT + (xcd * 8 + peer) * 128);
      asm volatile("global_load_dword %0, %1, off sc1" : "=v"(gp) : "v"(gw) : "memory");
      acc[7][3][3] = __int_as_float(max(__float_as_int(acc[7][3][3]), 0));
      u32x4 mk;
#pragma unroll
      for (int w = 0; w < 4; ++w) {
        unsigned m = 0;
#pragma unroll
        for (int i = 31; i >= 0; --i) m = __builtin_amdgcn_alignbit(m, __float_as_uint(acc[2 * w + (i >> 4)][(i >> 2) & 3][i & 3]), 31);
        mk[w] = ~m;
      }
      STORE_SYS(peerws + WS_MSK + (size_t)(par * 1024 + slot) * 1024 + lane * 16, mk);
      const int cnt = __popc(mk[0]) + __popc(mk[1]) + __popc(mk[2]) + __popc(mk[3]);
      const int inc = wave_scan(cnt);
      G = ((unsigned)__builtin_amdgcn_readlane(inc, 63) + 17u) / 18u;
      unsigned a = (unsigned)(size_t)lds + wid * L_STG + (inc - cnt) * 4;
#pragma unroll
      for (int i = 0; i < 32; ++i) {
        const f32x4 v = acc[i >> 2][i & 3];
        int sg;
        unsigned r1, r2, r3;
        asm volatile("v_ashrrev_i32 %1, 31, %5\n\tv_lshl_add_u32 %2, %1, 2, %0\n\tds_write_b32 %0, %5 offset:%9\n\t"
                     "v_ashrrev_i32 %1, 31, %6\n\tv_lshl_add_u32 %3, %1, 2, %2\n\tds_write_b32 %2, %6 offset:%9+4\n\t"
                     "v_ashrrev_i32 %1, 31, %7\n\tv_lshl_add_u32 %4, %1, 2, %3\n\tds_write_b32 %3, %7 offset:%9+8\n\t"
                     "v_ashrrev_i32 %1, 31, %8\n\tv_lshl_add_u32 %0, %1, 2, %4\n\tds_write_b32 %4, %8 offset:%9+12"
                     : "+v"(a), "=&v"(sg), "=&v"(r1), "=&v"(r2), "=&v"(r3) : "v"(v[0]), "v"(v[1]), "v"(v[2]), "v"(v[3]), "n"(16 * i) : "memory");
      }
      tk = (15u - ((drawn(tk) - 1u) & 15u)) * 8 + xcd;
      asm volatile("" : "+v"(gp));
      if (lane == 0) {
        __hip_atomic_store((GAS u64*)(peerws + WS_FLG + slot * 128), ((u64)call << 32) | tk, __ATOMIC_RELAXED, __HIP_MEMORY_SCOPE_SYSTEM);
        __hip_atomic_store((GAS unsigned*)gw, G, __ATOMIC_RELAXED, __HIP_MEMORY_SCOPE_AGENT);
      }
      Gs = max(G, min((unsigned)__builtin_amdgcn_readfirstlane(gp), 512u));
      const unsigned tag = (call & 2u) ? 0x00800000u : 0x00000080u;
#pragma unroll
      for (int j = 0; j < 8; ++j) {
        tg[j] = j * 64 + lane < G ? tag : 0u;
        if (j * 64 < G) {
#pragma unroll
          for (int k = 0; k < 4; ++k)
            acc[j][k] = *(LAS f32x4*)(lds + wid * L_STG + j * 4096 + (k * min(G - j * 64, 64u) + lane) * 16);
          ex[j] = *(LAS f32x2*)(lds + wid * L_STG + min(G * 64 + (j * 64 + lane) * 8, (unsigned)(L_STG - 8)));
        } else {
          ex[j] = (f32x2){0.f, 0.f};
        }
      }
      asm volatile("s_waitcnt lgkmcnt(0)" ::: "memory");
      if (lane == 0) __hip_atomic_store((LAS unsigned*)(lds + L_FREE + wid * 4), 1u, __ATOMIC_RELEASE, __HIP_MEMORY_SCOPE_WORKGROUP);
    }
    float inv; unsigned ts;
    if (wid == 0) {
      GAS u64* gl = (GAS u64*)(myws + WS_GLB + par * 128);
      int mv; bool bad;
      for (;;) {
        const u64 v = __hip_atomic_load(gl + (lane & 7), __ATOMIC_RELAXED, __HIP_MEMORY_SCOPE_SYSTEM);
        mv = (int)(unsigned)v;
        bad = !__all((unsigned)(v >> 32) == call);
        if (!bad || __builtin_amdgcn_s_memrealtime() - t0 > DEADLINE_TICKS) break;
        __builtin_amdgcn_s_sleep(1);
      }
      ts = ((unsigned)__builtin_amdgcn_s_memrealtime() << 2) | 1u;
      mv = max(mv, DPP0(mv, 0x111, 0xf)); mv = max(mv, DPP0(mv, 0x112, 0xf)); mv = max(mv, DPP0(mv, 0x114, 0xf));
      const float amax = __int_as_float(__builtin_amdgcn_readlane(mv, 7));
      const float scale = amax / 448.0f;
      inv = amax > 0.f ? 1.0f / scale : 0.f;
      if (bad) inv = __int_as_float(0x7fc00000);
      if (lane == 0)
        __hip_atomic_store((LAS u64*)(lds + L_SCALE), ((u64)ts << 32) | __float_as_uint(inv), __ATOMIC_RELEASE, __HIP_MEMORY_SCOPE_WORKGROUP);
    } else {
      u64 v;
      for (;;) {
        v = __hip_atomic_load((LAS u64*)(lds + L_SCALE), __ATOMIC_ACQUIRE, __HIP_MEMORY_SCOPE_WORKGROUP);
        ts = __builtin_amdgcn_readfirstlane((unsigned)(v >> 32));
        if (ts != 0u) break;
        if (__builtin_amdgcn_s_memrealtime() - t0 > 2 * DEADLINE_TICKS) { v = 0x7fc00000u; tk = 0; break; }
        __builtin_amdgcn_s_sleep(1);
      }
      inv = __uint_as_float((unsigned)v);
    }
    if (local) {
#pragma unroll
      for (int mi = 0; mi < 8; ++mi) {
        unsigned d[4];
#pragma unroll
        for (int ni = 0; ni < 4; ++ni) {
          f32x4 v;
#pragma unroll
          for (int r = 0; r < 4; ++r) v[r] = __int_as_float(max(__float_as_int(acc[mi][ni][r]), 0));
          d[ni] = pack4(v * inv);
        }
        u32x4 o;
        o[0] = odd ? d[1] : d[0]; o[1] = odd ? d[0] : d[1]; o[2] = odd ? d[3] : d[2]; o[3] = odd ? d[2] : d[3];
        STORE_OUT(mi, o);
      }
    } else {
      unsigned char* vd = peerws + WS_VAL + (size_t)(par * 1024 + slot) * 8192 + lane * 16;
      auto conv = [&](int j) {
        const f32x2 xe = ex[j] * inv;
        const unsigned e = (unsigned)__builtin_amdgcn_cvt_pk_fp8_f32(xe[0], xe[1], __float_as_int(xe[0]), false);
        const unsigned E = bfi(0x00800080u, tg[j], __builtin_amdgcn_perm(e << 4, e, 0x05010400u));
        u32x4 o;
#pragma unroll
        for (int k = 0; k < 4; ++k) o[k] = bfi(0x80808080u, E << k, pack4(acc[j][k] * inv));
        return o;
      };
      if (tk < 16) {
#pragma unroll
        for (int j = 0; j < 8; ++j)
          if (j * 64 < Gs) { const u32x4 o = conv(j); if (j * 64 + lane < Gs) STORE_SYS(vd + j * 1024, o); }
      } else {
        u32x4 o[8];
#pragma unroll
        for (int j = 0; j < 8; ++j) { if (j * 64 < Gs) o[j] = conv(j); else o[j] = (u32x4){0u, 0u, 0u, 0u}; asm volatile("" : "+v"(o[j])); }
        const int dt = (int)((tk * G * PACE) / 2580u);
        while ((int)(((unsigned)__builtin_amdgcn_s_memrealtime() << 2) - (ts & ~3u)) < dt * 4) __builtin_amdgcn_s_sleep(1);
#pragma unroll
        for (int j = 0; j < 8; ++j)
          if (j * 64 < Gs) { if (j * 64 + lane < Gs) STORE_SYS(vd + j * 1024, o[j]); }
      }
    }
  }
}

}

extern "C" void kernel_launch(int rank, int nranks, ncclComm_t comm, hipStream_t stream,
                              void* const* d_in, const int* in_sizes, int n_in,
                              void* d_out, int out_size, const int* in_dtypes, int out_dtype,
                              void* const* d_out_all, void* const* d_ws_all, size_t ws_bytes) {
  (void)comm; (void)in_dtypes; (void)out_dtype; (void)d_out_all;
  if (nranks != NR || n_in != 2 || in_sizes[0] != MLOC * KDIM || in_sizes[1] != KDIM * NDIM ||
      out_size != NR * MLOC * NSH || ws_bytes < WS_END || (unsigned)rank >= (unsigned)NR) {
    fprintf(stderr, "unexpected problem shape\n");
    return;
  }
  Params p;
  p.x = (const unsigned short*)d_in[0];
  p.w = (const unsigned short*)d_in[1];
  p.out = (unsigned char*)d_out;
  for (int i = 0; i < NR; ++i) p.ws[i] = (unsigned char*)d_ws_all[i];
  p.my = p.ws[rank]; p.rank = rank; p.pad = 0;
  hipLaunchKernelGGL(fused_gemm_a2a, dim3(NWG), dim3(512), 0, stream, p);
  HIP_CHECK(hipGetLastError());
}
```
